```python
import numpy as np

kernel_name = "gpu_ag_gemm_m4096_k4096_n16384_bf16_gelu"


ABSMAX_RTOL = 0.008

COMPUTE_DTYPE = "bfloat16"
EPILOGUE = "gelu"
IN_QUANT = None
OUT_QUANT = None
IN_DTYPES = {"x": "bfloat16", "w_mat": "bfloat16"}
WORLD_SIZE = 8

def _to_bf16(x):
    u = np.ascontiguousarray(x, dtype=np.float32).view(np.uint32)
    return ((u + 0x7FFF + ((u >> 16) & 1)) & 0xFFFF0000).view(np.float32)

def setup_inputs():

    x = _to_bf16(np.random.randn(4096, 4096).astype(np.float32))
    w_mat = _to_bf16((np.random.randn(4096, 16384) / np.sqrt(4096)).astype(np.float32))
    return {"x": x, "w_mat": w_mat}

def _epilogue(y):
    c = 0.7978845608028654
    return (0.5 * y * (1.0 + np.tanh(c * (y + 0.044715 * y ** 3)))).astype(np.float32)

def reference(x, w_mat):
    y = (x @ w_mat).astype(np.float32)
    return _epilogue(y)
```

```cpp
#include <hip/hip_runtime.h>
#include <rccl/rccl.h>
#include <cstdio>
#include <cstdint>

#define HIP_CHECK(expr) do { hipError_t _e = (expr); if (_e != hipSuccess) { \
    fprintf(stderr, "HIP error %s at %s:%d\n", hipGetErrorString(_e), __FILE__, __LINE__); \
    return; } } while (0)

typedef __attribute__((ext_vector_type(8))) short bf16x8;
typedef __attribute__((ext_vector_type(4))) short s16x4;
typedef __attribute__((ext_vector_type(4))) float f32x4;
typedef __attribute__((ext_vector_type(4))) unsigned u32x4;
typedef __attribute__((ext_vector_type(2))) unsigned u32x2;

constexpr int NR = 8;
constexpr int GM = 4096, GK = 4096, NL = 2048, MS = 512;
constexpr int BM = 256, BN = 128, BK = 64, NT = GK / BK;
constexpr int NCH = 16, KC = GK / NCH, IPC = KC / BK;
constexpr int NWG = 256;
constexpr int NSTAGE = 3;
constexpr int A_BYTES = BM * BK * 2, B_BYTES = BK * BN * 2, ST_BYTES = A_BYTES + B_BYTES;
constexpr int CTRL_OFF = NSTAGE * ST_BYTES;
constexpr int LDS_BYTES = CTRL_OFF + 64;

constexpr size_t XBUF_BYTES = (size_t)GM * GK * 2;
constexpr size_t WS_CTRL = 2 * XBUF_BYTES;
constexpr size_t WS_FLAGS = WS_CTRL;
constexpr size_t WS_TICK = WS_CTRL + 16384;
constexpr size_t WS_TMO = WS_CTRL + 24576;
constexpr size_t WS_CALL = WS_CTRL + 32768;
constexpr size_t WS_NEED = WS_CALL + (size_t)NWG * 128;
constexpr unsigned long long DEADLINE_TICKS = 50000000ull;

struct Args {
  const char* x; const char* w; char* out; char* ws[NR]; int rank;
};

__device__ __forceinline__ int ldsA_byte(int r, int c) {
  int st = (r >> 4) * 2 + (c >> 5), ob = (r & 15) * 64 + (c & 31) * 2;
  return st * 1024 + (ob ^ (((ob >> 9) & 1) << 5));
}
__device__ __forceinline__ int swzB(int row) { return ((row & 3) << 2) | ((row >> 2) & 3); }

__device__ __forceinline__ float gelu_f(float y) {
  const float c2 = 2.0f * 0.7978845608028654f;
  float u = c2 * (y + 0.044715f * y * y * y);
  return y * __builtin_amdgcn_rcpf(1.0f + __expf(-u));
}
__device__ __forceinline__ unsigned pack_bf16(float a, float b) {
  typedef __attribute__((ext_vector_type(2))) __bf16 b2;
  b2 v; v[0] = (__bf16)a; v[1] = (__bf16)b;
  return __builtin_bit_cast(unsigned, v);
}

__global__ __launch_bounds__(576, 1) void ag_gemm_kernel(Args a) {
  extern __shared__ __attribute__((aligned(16))) char smem[];
  const int tid = threadIdx.x, lane = tid & 63;
  const int wid = __builtin_amdgcn_readfirstlane(tid >> 6);
  const int b = blockIdx.x, rank = a.rank;
  char* myws = a.ws[rank];
  volatile unsigned* ctl = (volatile unsigned*)(smem + CTRL_OFF);
  if (tid == 0) {
    unsigned* cp = (unsigned*)(myws + WS_CALL + (size_t)b * 128);
    unsigned c = __hip_atomic_load(cp, __ATOMIC_RELAXED, __HIP_MEMORY_SCOPE_AGENT) + 1u;
    __hip_atomic_store(cp, c, __ATOMIC_RELAXED, __HIP_MEMORY_SCOPE_AGENT);
    ctl[0] = 0; ctl[1] = 0; ctl[2] = c;
  }
  __syncthreads();
  const unsigned call = __builtin_amdgcn_readfirstlane(ctl[2]);
  const size_t xoff = (call & 1u) ? XBUF_BYTES : 0;

  if (wid == 8) {
    const int row = 2 * b + (lane >> 5);
    const size_t soff = (size_t)row * (GK * 2) + (size_t)(lane & 31) * 16;
    const size_t doff = xoff + (size_t)rank * MS * (GK * 2) + soff;
    u32x4 v[NCH];
#pragma unroll
    for (int c = 0; c < NCH; ++c) v[c] = *(const u32x4*)(a.x + soff + (size_t)c * (KC * 2));
#pragma unroll
    for (int c = 0; c < NCH; ++c) {
#pragma unroll
      for (int j = 1; j < NR; ++j) {
        int p = (rank + 1 + ((j - 1 + b) % 7)) & 7;
        *(u32x4*)(a.ws[p] + doff + (size_t)c * (KC * 2)) = v[c];
      }
      asm volatile("s_waitcnt vmcnt(0)" ::: "memory");
      unsigned t = 0;
      if (lane == 0)
        t = __hip_atomic_fetch_add((unsigned*)(myws + WS_TICK + (size_t)c * 128), 1u, __ATOMIC_RELEASE, __HIP_MEMORY_SCOPE_AGENT);
      t = __builtin_amdgcn_readfirstlane(t);
      if ((t & (NWG - 1)) == NWG - 1) {
        __builtin_amdgcn_fence(__ATOMIC_ACQUIRE, "agent");
        __builtin_amdgcn_fence(__ATOMIC_RELEASE, "");
        asm volatile("s_waitcnt vmcnt(0)" ::: "memory");
        if (lane < NR && lane != rank) {
          char* pw = a.ws[0];
#pragma unroll
          for (int p = 1; p < NR; ++p) pw = (lane == p) ? a.ws[p] : pw;
          __hip_atomic_store((unsigned*)(pw + WS_FLAGS + (size_t)(rank * NCH + c) * 128), call, __ATOMIC_RELAXED, __HIP_MEMORY_SCOPE_SYSTEM);
        }
      }
    }
    return;
  }

  const int xcd = b & 7, idx = b >> 3;
  const int mblk = (xcd >> 1) * 4 + (idx & 3), nblk = (xcd & 1) * 8 + (idx >> 2);
  const int src = mblk >> 1;
  const bool local = (src == rank);
  const char* Abase = local ? (a.x + (size_t)(mblk & 1) * BM * (GK * 2)) : (myws + xoff + (size_t)mblk * BM * (GK * 2));
  const char* Bbase = a.w + (size_t)nblk * BN * 2;
  unsigned* flagp = (unsigned*)(myws + WS_FLAGS + (size_t)src * NCH * 128);
  unsigned* tmop = (unsigned*)(myws + WS_TMO);

  unsigned aoff[4], boff[2];
#pragma unroll
  for (int i = 0; i < 4; ++i) {
    int bb = (wid * 4 + i) * 1024 + lane * 16;
    int st = bb >> 10, sb = bb & 1023, sw = sb ^ (((sb >> 9) & 1) << 5);
    int R = (st >> 1) * 16 + (sw >> 6), C = (st & 1) * 32 + ((sw & 63) >> 1);
    aoff[i] = (unsigned)(R * (GK * 2) + C * 2);
  }
#pragma unroll
  for (int i = 0; i < 2; ++i) {
    int bb = (wid * 2 + i) * 1024 + lane * 16;
    int row = bb >> 8, pos = (bb >> 4) & 15, ch = pos ^ swzB(row);
    boff[i] = (unsigned)(row * (NL * 2) + ch * 16);
  }
  auto stage = [&](int t, int s) {
    char* la = smem + s * ST_BYTES + wid * 4096;
    char* lb = smem + s * ST_BYTES + A_BYTES + wid * 2048;
    const char* ga = Abase + (size_t)t * (BK * 2);
    const char* gb = Bbase + (size_t)t * BK * (NL * 2);
#pragma unroll
    for (int i = 0; i < 4; ++i)
      __builtin_amdgcn_global_load_lds((const unsigned*)(ga + aoff[i]), (unsigned*)(la + i * 1024 + lane * 16), 16, 0, 0);
#pragma unroll
    for (int i = 0; i < 2; ++i)
      __builtin_amdgcn_global_load_lds((const unsigned*)(gb + boff[i]), (unsigned*)(lb + i * 1024 + lane * 16), 16, 0, 0);
  };
  auto wait_chunk = [&](int c) {
    unsigned r = __builtin_amdgcn_readfirstlane(ctl[1]);
    if (r > (unsigned)c) return;
    unsigned long long t0 = __builtin_amdgcn_s_memrealtime();
    while (__hip_atomic_load(flagp + c * 32, __ATOMIC_RELAXED, __HIP_MEMORY_SCOPE_SYSTEM) < call) {
      if (__hip_atomic_load(tmop, __ATOMIC_RELAXED, __HIP_MEMORY_SCOPE_AGENT) == call ||
          __builtin_amdgcn_s_memrealtime() - t0 > DEADLINE_TICKS) {
        __hip_atomic_store(tmop, call, __ATOMIC_RELAXED, __HIP_MEMORY_SCOPE_AGENT);
        break;
      }
      __builtin_amdgcn_s_sleep(2);
    }
    __builtin_amdgcn_fence(__ATOMIC_ACQUIRE, "");
    asm volatile("s_waitcnt vmcnt(0)" ::: "memory");
    if (lane == 0) __hip_atomic_fetch_max((unsigned*)(smem + CTRL_OFF + 4), (unsigned)(c + 1), __ATOMIC_RELAXED, __HIP_MEMORY_SCOPE_WORKGROUP);
  };

  const int wm = wid >> 1, wn = wid & 1, fr = lane & 15, fq = lane >> 4;
  int ard[4][2], brd[2][2];
#pragma unroll
  for (int mi = 0; mi < 4; ++mi)
#pragma unroll
    for (int kk = 0; kk < 2; ++kk) ard[mi][kk] = ldsA_byte(wm * 64 + mi * 16 + fr, kk * 32 + fq * 8);
  const int q = fr >> 2, p = lane & 3;
  f32x4 acc[4][4];
#pragma unroll
  for (int i = 0; i < 4; ++i)
#pragma unroll
    for (int j = 0; j < 4; ++j) acc[i][j] = (f32x4){0.f, 0.f, 0.f, 0.f};
  (void)brd;

  if (!local) wait_chunk(0);
  stage(0, 0);
  stage(1, 1);
  int s = 0;
  for (int t = 0; t < NT; ++t) {
    if (t + 1 < NT) asm volatile("s_waitcnt vmcnt(6)" ::: "memory");
    else asm volatile("s_waitcnt vmcnt(0)" ::: "memory");
    if (lane == 0) __hip_atomic_fetch_add((unsigned*)(smem + CTRL_OFF), 1u, __ATOMIC_RELAXED, __HIP_MEMORY_SCOPE_WORKGROUP);
    const unsigned target = 8u * (unsigned)(t + 1);
    while (__builtin_amdgcn_readfirstlane(ctl[0]) < target) { __builtin_amdgcn_s_sleep(0); }
    asm volatile("" ::: "memory");
    if (t + 2 < NT) {
      if (!local && ((t + 2) % IPC) == 0) wait_chunk((t + 2) / IPC);
      int s2 = s + 2; if (s2 >= NSTAGE) s2 -= NSTAGE;
      stage(t + 2, s2);
    }
    const char* sa = smem + s * ST_BYTES;
    const char* sb = sa + A_BYTES;
#pragma unroll
    for (int kk = 0; kk < 2; ++kk) {
      bf16x8 af[4], bfr[4];
#pragma unroll
      for (int ni = 0; ni < 4; ++ni) {
        s16x4 h[2];
#pragma unroll
        for (int tt = 0; tt < 2; ++tt) {
          int row = 32 * kk + 8 * fq + 4 * tt + q;
          int ch = 2 * (wn * 4 + ni) + (p >> 1);
          int off = 256 * row + 16 * (ch ^ swzB(row)) + 8 * (p & 1);
          h[tt] = __builtin_amdgcn_ds_read_tr16_b64_v4i16((s16x4 __attribute__((address_space(3)))*)(sb + off));
        }
        bfr[ni] = __builtin_shufflevector(h[0], h[1], 0, 1, 2, 3, 4, 5, 6, 7);
      }
#pragma unroll
      for (int mi = 0; mi < 4; ++mi) af[mi] = *(const bf16x8*)(sa + ard[mi][kk]);
#pragma unroll
      for (int mi = 0; mi < 4; ++mi)
#pragma unroll
        for (int ni = 0; ni < 4; ++ni)
          acc[mi][ni] = __builtin_amdgcn_mfma_f32_16x16x32_bf16(bfr[ni], af[mi], acc[mi][ni], 0, 0, 0);
    }
    asm volatile("" ::: "memory");
    s = s + 1; if (s >= NSTAGE) s = 0;
  }

  const bool bad = (__hip_atomic_load(tmop, __ATOMIC_RELAXED, __HIP_MEMORY_SCOPE_AGENT) == call);
  char* ob = a.out + ((size_t)(mblk * BM + wm * 64 + fr) * NL + nblk * BN + wn * 64 + fq * 4) * 2;
#pragma unroll
  for (int mi = 0; mi < 4; ++mi)
#pragma unroll
    for (int ni = 0; ni < 4; ++ni) {
      u32x2 o;
      o[0] = pack_bf16(gelu_f(acc[mi][ni][0]), gelu_f(acc[mi][ni][1]));
      o[1] = pack_bf16(gelu_f(acc[mi][ni][2]), gelu_f(acc[mi][ni][3]));
      if (bad) { o[0] = 0x7FC07FC0u; o[1] = 0x7FC07FC0u; }
      *(u32x2*)(ob + ((size_t)mi * 16 * NL + ni * 16) * 2) = o;
    }
}

extern "C" void kernel_launch(int rank, int nranks, ncclComm_t comm,
                              hipStream_t stream,
                              void* const* d_in, const int* in_sizes, int n_in,
                              void* d_out, int out_size,
                              const int* in_dtypes, int out_dtype,
                              void* const* d_out_all,
                              void* const* d_ws_all, size_t ws_bytes) {
  if (nranks != NR || ws_bytes < WS_NEED || (unsigned)rank >= (unsigned)NR) { fprintf(stderr, "unexpected configuration\n"); return; }
  static bool init = false;
  if (!init) {
    HIP_CHECK(hipFuncSetAttribute((const void*)ag_gemm_kernel, hipFuncAttributeMaxDynamicSharedMemorySize, LDS_BYTES));
    init = true;
  }
  Args a;
  a.x = (const char*)d_in[0]; a.w = (const char*)d_in[1]; a.out = (char*)d_out; a.rank = rank;
  for (int p = 0; p < NR; ++p) a.ws[p] = (char*)d_ws_all[p];
  ag_gemm_kernel<<<NWG, 576, LDS_BYTES, stream>>>(a);
  HIP_CHECK(hipGetLastError());
}
```

```cpp
#include <hip/hip_runtime.h>
#include <rccl/rccl.h>
#include <cstdio>
#include <cstdint>

#define HIP_CHECK(expr) do { hipError_t _e = (expr); if (_e != hipSuccess) { \
    fprintf(stderr, "HIP error %s at %s:%d\n", hipGetErrorString(_e), __FILE__, __LINE__); \
    return; } } while (0)

typedef __attribute__((ext_vector_type(8))) short bf16x8;
typedef __attribute__((ext_vector_type(4))) short s16x4;
typedef __attribute__((ext_vector_type(4))) float f32x4;
typedef __attribute__((ext_vector_type(4))) unsigned u32x4;
typedef __attribute__((ext_vector_type(2))) unsigned u32x2;

constexpr int NR = 8;
constexpr int GM = 4096, GK = 4096, NL = 2048, MS = 512;
constexpr int BM = 256, BN = 128, BK = 64, NT = GK / BK;
#define NCHUNK 32
constexpr int NCH = NCHUNK, KC = GK / NCH, IPC = KC / BK;
constexpr int NWG = 256, NCW = 16;
constexpr int NSTAGE = 3;
constexpr int A_BYTES = BM * BK * 2, B_BYTES = BK * BN * 2, ST_BYTES = A_BYTES + B_BYTES;
constexpr int CTRL_OFF = NSTAGE * (ST_BYTES + 4096);
constexpr int LDS_BYTES = CTRL_OFF + 64;

constexpr size_t XBUF_BYTES = (size_t)GM * GK * 2;
constexpr size_t WS_CTRL = 2 * XBUF_BYTES;
constexpr size_t WS_FLAGS = WS_CTRL;
constexpr size_t WS_TICK = WS_CTRL + 16384;
constexpr size_t WS_TMO = WS_CTRL + 24576;
constexpr size_t WS_HEAD = WS_CTRL + 24576 + 1024;
#define INFL 7
constexpr size_t WS_CALL = WS_CTRL + 32768;
constexpr size_t WS_DBG = WS_CALL + (size_t)NWG * 128;
constexpr size_t WS_NEED = WS_DBG + (size_t)NWG * 64;
#define TS(k) do {} while (0)
constexpr unsigned long long DEADLINE_TICKS = 50000000ull;

struct Args {
  const char* x; const char* w; char* out; char* ws[NR]; int rank;
};

__device__ __forceinline__ int ldsA_byte(int r, int c) {
  int st = (r >> 4) * 2 + (c >> 5), ob = (r & 15) * 64 + (c & 31) * 2;
  return st * 1024 + (ob ^ (((ob >> 9) & 1) << 5));
}
__device__ __forceinline__ int swzB(int row) { return ((row & 3) << 2) | ((row >> 2) & 3); }

__device__ __forceinline__ float gelu_f(float y) {
  const float c2 = 2.0f * 0.7978845608028654f;
  float u = c2 * (y + 0.044715f * y * y * y);
  return y * __builtin_amdgcn_rcpf(1.0f + __expf(-u));
}
__device__ __forceinline__ unsigned pack_bf16(float a, float b) {
  typedef __attribute__((ext_vector_type(2))) __bf16 b2;
  b2 v; v[0] = (__bf16)a; v[1] = (__bf16)b;
  return __builtin_bit_cast(unsigned, v);
}

#define LDS_ADDR(p) ((unsigned)(size_t)(__attribute__((address_space(3))) char*)(p))

template <int MI, int EX>
__device__ __forceinline__ void run_tile(char* smem, const char* Abase, const char* Bbase, char* obase, const char* Xbase, char* xobase, bool gated,
                                         unsigned* flagp, unsigned* tmop, unsigned call, unsigned cnt0, int wid, int lane
                                         ) {
#define MSP 2
  constexpr int MSPLIT = MI == 1 ? 0 : MSP;
  constexpr int NLD = MI + 2 + EX;
  constexpr int AB = MI * 8192 + EX * 4096, STB = AB + B_BYTES, NS = CTRL_OFF / STB, D = NS - 1;
  unsigned aoff[MI], boff[2];
#pragma unroll
  for (int i = 0; i < MI; ++i) {
    int bb = (wid * MI + i) * 1024 + lane * 16;
    int st = bb >> 10, sb = bb & 1023, sw = sb ^ (((sb >> 9) & 1) << 5);
    int R = (st >> 1) * 16 + (sw >> 6), C = (st & 1) * 32 + ((sw & 63) >> 1);
    aoff[i] = (unsigned)(R * (GK * 2) + C * 2);
  }
#pragma unroll
  for (int i = 0; i < 2; ++i) {
    int bb = (wid * 2 + i) * 1024 + lane * 16;
    int row = bb >> 8, pos = (bb >> 4) & 15, ch = pos ^ swzB(row);
    boff[i] = (unsigned)(row * (NL * 2) + ch * 16);
  }
  unsigned xo = 0;
  if (EX) {
    int bb = wid * 512 + (lane & 31) * 16;
    int st = bb >> 10, sb = bb & 1023, sw = sb ^ (((sb >> 9) & 1) << 5);
    int R = (st >> 1) * 16 + (sw >> 6), C = (st & 1) * 32 + ((sw & 63) >> 1);
    xo = (unsigned)(R * (GK * 2) + C * 2);
  }
  auto stage = [&](int t, int s) {
    char* la = smem + s * STB + wid * (MI * 1024);
    char* lb = smem + s * STB + AB + wid * 2048;
    const char* ga = Abase + (size_t)t * (BK * 2);
    const char* gb = Bbase + (size_t)t * BK * (NL * 2);
#pragma unroll
    for (int i = 0; i < MI; ++i)
      __builtin_amdgcn_global_load_lds((const unsigned*)(ga + aoff[i]), (unsigned*)(la + i * 1024 + lane * 16), 16, 0, 0);
#pragma unroll
    for (int i = 0; i < 2; ++i)
      __builtin_amdgcn_global_load_lds((const unsigned*)(gb + boff[i]), (unsigned*)(lb + i * 1024 + lane * 16), 16, 0, 0);
    if (EX && lane < 32)
      __builtin_amdgcn_global_load_lds((const unsigned*)(Xbase + (size_t)t * (BK * 2) + xo), (unsigned*)(smem + s * STB + MI * 8192 + wid * 512 + lane * 16), 16, 0, 0);
  };
  const unsigned lds0 = LDS_ADDR(smem);
  const unsigned cnt_addr = lds0 + CTRL_OFF;
  unsigned rdy = 0;
  auto wait_chunk = [&](int c) {
    if (rdy > (unsigned)c) return;
    if (wid != 0) {
      for (;;) {
        unsigned r;
        asm volatile("ds_read_b32 %0, %1 offset:4\n\ts_waitcnt lgkmcnt(0)" : "=v"(r) : "v"(cnt_addr) : "memory");
        rdy = __builtin_amdgcn_readfirstlane(r);
        if (rdy > (unsigned)c) break;
        __builtin_amdgcn_s_sleep(1);
      }
      return;
    }
    unsigned long long t0 = __builtin_amdgcn_s_memrealtime();
    unsigned nr;
    for (;;) {
      unsigned f = __hip_atomic_load(flagp + (lane & (NCH - 1)), __ATOMIC_RELAXED, __HIP_MEMORY_SCOPE_SYSTEM);
      unsigned long long m = ~__builtin_amdgcn_ballot_w64(f >= call);
      nr = m ? (unsigned)__builtin_ctzll(m) : 64u;
      if (nr > (unsigned)c) break;
      if (__hip_atomic_load(tmop, __ATOMIC_RELAXED, __HIP_MEMORY_SCOPE_AGENT) == call ||
          __builtin_amdgcn_s_memrealtime() - t0 > DEADLINE_TICKS) {
        __hip_atomic_store(tmop, call, __ATOMIC_RELAXED, __HIP_MEMORY_SCOPE_AGENT);
        nr = 64u;
        break;
      }
      __builtin_amdgcn_s_sleep(2);
    }
    if (lane == 0) asm volatile("ds_write_b32 %0, %1 offset:4" :: "v"(cnt_addr), "v"(nr) : "memory");
    rdy = nr;
    if (c == 0) TS(2);
    if (c <= NCH / 2) TS(5);
    TS(6);
  };

  const int wm = wid >> 1, wn = wid & 1, fr = lane & 15, fq = lane >> 4;
  const int q = fr >> 2, p = lane & 3;
  const unsigned abase = lds0 + (unsigned)ldsA_byte(wm * (16 * MI) + fr, fq * 8);
  const unsigned ebase = lds0 + (unsigned)ldsA_byte(64 * MI + (wm >> 1) * 16 + fr, fq * 8);
  f32x4 eacc[2] = {(f32x4){0.f, 0.f, 0.f, 0.f}, (f32x4){0.f, 0.f, 0.f, 0.f}};
  unsigned bbase[4][2];
#pragma unroll
  for (int ni = 0; ni < 4; ++ni)
#pragma unroll
    for (int tt = 0; tt < 2; ++tt) {
      int row = 8 * fq + 4 * tt + q, ch = 2 * (wn * 4 + ni) + (p >> 1);
      bbase[ni][tt] = lds0 + AB + (unsigned)(256 * row + 16 * (ch ^ swzB(row)) + 8 * (p & 1));
    }
  f32x4 acc[MI][4];
#pragma unroll
  for (int i = 0; i < MI; ++i)
#pragma unroll
    for (int j = 0; j < 4; ++j) acc[i][j] = (f32x4){0.f, 0.f, 0.f, 0.f};

  bf16x8 af[2][MI], ef[2]; s16x4 bh[2][4][2];
  auto read_frags = [&](int kk, unsigned so) {
#pragma unroll
    for (int ni = 0; ni < 4; ++ni)
#pragma unroll
      for (int tt = 0; tt < 2; ++tt) {
        unsigned ba = bbase[ni][tt] + so;
        if (kk == 0) asm volatile("ds_read_b64_tr_b16 %0, %1" : "=v"(bh[kk][ni][tt]) : "v"(ba) : "memory");
        else asm volatile("ds_read_b64_tr_b16 %0, %1 offset:8192" : "=v"(bh[kk][ni][tt]) : "v"(ba) : "memory");
      }
#pragma unroll
    for (int mi = 0; mi < MI; ++mi)
      asm volatile("ds_read_b128 %0, %1 offset:%2" : "=v"(af[kk][mi]) : "v"(abase + so), "i"(mi * 2048 + kk * 1024) : "memory");
    if (EX) asm volatile("ds_read_b128 %0, %1 offset:%2" : "=v"(ef[kk]) : "v"(ebase + so), "i"(kk * 1024) : "memory");
  };
  auto pin = [&](int kk) {
#pragma unroll
    for (int mi = 0; mi < MI; ++mi) asm volatile("" : "+v"(af[kk][mi]));
#pragma unroll
    for (int ni = 0; ni < 4; ++ni) asm volatile("" : "+v"(bh[kk][ni][0]), "+v"(bh[kk][ni][1]));
    if (EX) asm volatile("" : "+v"(ef[kk]));
  };
  auto mma = [&](int kk, int m0, int m1) {
#pragma unroll
    for (int mi = m0; mi < m1; ++mi)
#pragma unroll
      for (int ni = 0; ni < 4; ++ni)
        acc[mi][ni] = __builtin_amdgcn_mfma_f32_16x16x32_bf16(__builtin_shufflevector(bh[kk][ni][0], bh[kk][ni][1], 0, 1, 2, 3, 4, 5, 6, 7), af[kk][mi], acc[mi][ni], 0, 0, 0);
    if (EX && m1 == MI) {
      if (wm & 1) {
        eacc[0] = __builtin_amdgcn_mfma_f32_16x16x32_bf16(__builtin_shufflevector(bh[kk][2][0], bh[kk][2][1], 0, 1, 2, 3, 4, 5, 6, 7), ef[kk], eacc[0], 0, 0, 0);
        eacc[1] = __builtin_amdgcn_mfma_f32_16x16x32_bf16(__builtin_shufflevector(bh[kk][3][0], bh[kk][3][1], 0, 1, 2, 3, 4, 5, 6, 7), ef[kk], eacc[1], 0, 0, 0);
      } else {
        eacc[0] = __builtin_amdgcn_mfma_f32_16x16x32_bf16(__builtin_shufflevector(bh[kk][0][0], bh[kk][0][1], 0, 1, 2, 3, 4, 5, 6, 7), ef[kk], eacc[0], 0, 0, 0);
        eacc[1] = __builtin_amdgcn_mfma_f32_16x16x32_bf16(__builtin_shufflevector(bh[kk][1][0], bh[kk][1][1], 0, 1, 2, 3, 4, 5, 6, 7), ef[kk], eacc[1], 0, 0, 0);
      }
    }
    __builtin_amdgcn_sched_barrier(0);
  };
  auto arrive = [&]() { if (lane == 0) asm volatile("ds_add_u32 %0, %1" :: "v"(cnt_addr), "v"(1u) : "memory"); };
  auto sync_stage = [&](int t) {
    const unsigned target = cnt0 + 8u * (unsigned)(t + 1);
    for (;;) {
      unsigned v;
      asm volatile("ds_read_b32 %0, %1\n\ts_waitcnt lgkmcnt(0)" : "=v"(v) : "v"(cnt_addr) : "memory");
      if (__builtin_amdgcn_readfirstlane(v) >= target) break;
    }
  };

#pragma unroll
  for (int i = 0; i < D; ++i) {
    if (gated && (i % IPC) == 0) wait_chunk(i / IPC);
    stage(i, i);
  }
  asm volatile("s_waitcnt vmcnt(%0)" :: "i"(NLD * (D - 1)) : "memory");
  arrive();
  int s = 0;
  for (int t = 0; t < NT; ++t) {
    sync_stage(t);
    if (t + D < NT) {
      if (gated && ((t + D) % IPC) == 0) wait_chunk((t + D) / IPC);
      int s2 = s + D; if (s2 >= NS) s2 -= NS;
      stage(t + D, s2);
    }
    read_frags(0, (unsigned)(s * STB));
    read_frags(1, (unsigned)(s * STB));
    asm volatile("s_waitcnt lgkmcnt(%0)" :: "i"(8 + MI + EX) : "memory");
    pin(0);
    mma(0, 0, MI);
    asm volatile("s_waitcnt lgkmcnt(0)" ::: "memory");
    pin(1);
    if (t + D < NT) asm volatile("s_waitcnt vmcnt(%0)" :: "i"(NLD * (D - 1)) : "memory");
    else if (D > 2 && t + D == NT) asm volatile("s_waitcnt vmcnt(%0)" :: "i"(NLD * (D > 2 ? D - 2 : 0)) : "memory");
    else if (D > 3 && t + D == NT + 1) asm volatile("s_waitcnt vmcnt(%0)" :: "i"(NLD * (D > 3 ? D - 3 : 0)) : "memory");
    else if (D > 4 && t + D == NT + 2) asm volatile("s_waitcnt vmcnt(%0)" :: "i"(NLD * (D > 4 ? D - 4 : 0)) : "memory");
    else asm volatile("s_waitcnt vmcnt(0)" ::: "memory");
    arrive();
    mma(1, 0, MI);
    s = s + 1; if (s >= NS) s = 0;
  }
  if (wid == 0 && gated) TS(3);

  const bool bad = gated && (__hip_atomic_load(tmop, __ATOMIC_RELAXED, __HIP_MEMORY_SCOPE_AGENT) == call);
  char* ob = obase + ((size_t)(wm * (16 * MI) + fr) * NL + wn * 64 + fq * 4) * 2;
#pragma unroll
  for (int mi = 0; mi < MI; ++mi)
#pragma unroll
    for (int ni = 0; ni < 4; ++ni) {
      u32x2 o;
      o[0] = pack_bf16(gelu_f(acc[mi][ni][0]), gelu_f(acc[mi][ni][1]));
      o[1] = pack_bf16(gelu_f(acc[mi][ni][2]), gelu_f(acc[mi][ni][3]));
      if (bad) { o[0] = 0x7FC07FC0u; o[1] = 0x7FC07FC0u; }
      *(u32x2*)(ob + ((size_t)mi * 16 * NL + ni * 16) * 2) = o;
    }
  if (EX) {
    char* xb = xobase + ((size_t)((wm >> 1) * 16 + fr) * NL + wn * 64 + (wm & 1) * 32 + fq * 4) * 2;
#pragma unroll
    for (int j = 0; j < 2; ++j) {
      u32x2 o;
      o[0] = pack_bf16(gelu_f(eacc[j][0]), gelu_f(eacc[j][1]));
      o[1] = pack_bf16(gelu_f(eacc[j][2]), gelu_f(eacc[j][3]));
      *(u32x2*)(xb + j * 32) = o;
    }
  }
}

__global__ __launch_bounds__(512, 1) void ag_gemm_kernel(Args a) {
  extern __shared__ __attribute__((aligned(16))) char smem[];
  const int tid = threadIdx.x, lane = tid & 63;
  const int wid = __builtin_amdgcn_readfirstlane(tid >> 6);
  const int b = blockIdx.x, rank = a.rank;
  char* myws = a.ws[rank];
  volatile unsigned* ctl = (volatile unsigned*)(smem + CTRL_OFF);
  if (tid == 0) {
    unsigned* cp = (unsigned*)(myws + WS_CALL + (size_t)b * 128);
    unsigned c = __hip_atomic_load(cp, __ATOMIC_RELAXED, __HIP_MEMORY_SCOPE_AGENT) + 1u;
    __hip_atomic_store(cp, c, __ATOMIC_RELAXED, __HIP_MEMORY_SCOPE_AGENT);
    ctl[0] = 0; ctl[1] = 0; ctl[2] = c;
    __builtin_amdgcn_fence(__ATOMIC_ACQUIRE, "");
    asm volatile("s_waitcnt vmcnt(0)" ::: "memory");
  }
  __syncthreads();
  const unsigned call = __builtin_amdgcn_readfirstlane(ctl[2]);
  const size_t xoff = (call & 1u) ? XBUF_BYTES : 0;
  const int xcd = b & 7, idx = b >> 3;
  if (wid == 0) TS(0);

  if (idx < NCW / 8) {
    constexpr unsigned RPI = 512 / KC, LPR = KC / 8;
    constexpr unsigned IPCH = MS / RPI, NITEMS = NCH * IPCH;
    const unsigned base = (call - 1u) * (NITEMS + NCW * 8);
    char* headp = myws + WS_HEAD;
    const size_t dbase = xoff + (size_t)rank * MS * (GK * 2);
    const unsigned lo = (unsigned)(lane / LPR) * (GK * 2) + (unsigned)(lane % LPR) * 16;
    char* pw = a.ws[0];
#pragma unroll
    for (int p = 1; p < NR; ++p) pw = (lane == p) ? a.ws[p] : pw;
    pw += WS_FLAGS + (size_t)rank * 256;
    char* dst[NR - 1];
#pragma unroll
    for (int j = 0; j < NR - 1; ++j) dst[j] = a.ws[(rank + 1 + (j + b * 8 + wid) % 7) & 7] + dbase;
    unsigned g = 0, tk = 0, pc = 0xffffffffu;
    u32x4 v0;
    if (lane == 0) asm volatile("global_atomic_add %0, %1, %2, off sc0" : "=v"(g) : "v"(headp), "v"(1u) : "memory");
    asm volatile("s_waitcnt vmcnt(0)" : "+v"(g) :: "memory");
    unsigned it = __builtin_amdgcn_readfirstlane(g) - base;
    unsigned off = (it & (IPCH - 1)) * (RPI * GK * 2) + (it / IPCH) * (KC * 2) + lo;
    if (it < NITEMS) asm volatile("global_load_dwordx4 %0, %1, off" : "=v"(v0) : "v"(a.x + off) : "memory");
    while (it < NITEMS) {
      if (lane == 0) asm volatile("global_atomic_add %0, %1, %2, off sc0" : "=v"(g) : "v"(headp), "v"(1u) : "memory");
      asm volatile("s_waitcnt vmcnt(1)" : "+v"(v0), "+v"(tk) :: "memory");
      asm volatile("global_store_dwordx4 %0, %1, off sc0 sc1" :: "v"(dst[0] + off), "v"(v0) : "memory");
      if (pc != 0xffffffffu && (__builtin_amdgcn_readfirstlane(tk) & (IPCH - 1)) == IPCH - 1 && lane < NR && lane != rank)
        asm volatile("global_store_dword %0, %1, off sc0 sc1" :: "v"(pw + (size_t)pc * 4), "v"(call) : "memory");
#pragma unroll
      for (int j = 1; j < NR - 1; ++j) {
        asm volatile("s_waitcnt vmcnt(%0)" :: "i"(INFL - 1) : "memory");
        asm volatile("global_store_dwordx4 %0, %1, off sc0 sc1" :: "v"(dst[j] + off), "v"(v0) : "memory");
      }
      asm volatile("s_waitcnt vmcnt(6)" : "+v"(g) :: "memory");
      pc = it / IPCH;
      it = __builtin_amdgcn_readfirstlane(g) - base;
      off = (it & (IPCH - 1)) * (RPI * GK * 2) + (it / IPCH) * (KC * 2) + lo;
      asm volatile("s_waitcnt vmcnt(0)" ::: "memory");
      if (lane == 0) asm volatile("global_atomic_add %0, %1, %2, off sc0" : "=v"(tk) : "v"(myws + WS_TICK + (size_t)pc * 128), "v"(1u) : "memory");
      if (it < NITEMS) asm volatile("global_load_dwordx4 %0, %1, off" : "=v"(v0) : "v"(a.x + off) : "memory");
    }
    asm volatile("s_waitcnt vmcnt(0)" : "+v"(tk) :: "memory");
    if (pc != 0xffffffffu && (__builtin_amdgcn_readfirstlane(tk) & (IPCH - 1)) == IPCH - 1 && lane < NR && lane != rank)
      asm volatile("global_store_dword %0, %1, off sc0 sc1" :: "v"(pw + (size_t)pc * 4), "v"(call) : "memory");
    if (wid == 0) TS(1);
    return;
  }

  unsigned* tmop = (unsigned*)(myws + WS_TMO);
#define TSARGS
  if (idx < NCW / 8 + 2) {
    const int nblk = xcd * 2 + (idx - NCW / 8);
    run_tile<1, 0>(smem, a.x + (size_t)448 * (GK * 2), a.w + (size_t)nblk * BN * 2, a.out + ((size_t)(rank * MS + 448) * NL + nblk * BN) * 2,
                   nullptr, nullptr, false, nullptr, tmop, call, 0u, wid, lane TSARGS);
    if (wid == 0) TS(7);
    return;
  }
  const int gi = idx - (NCW / 8 + 2);
  const int mi7 = gi % 7, nblk = (xcd & 3) * 4 + gi / 7;
  const int mg = (xcd >> 2) * 7 + mi7;
  const int mblk = (2 * (rank + 1) + mg) & 15;
  const int src = mblk >> 1;
  unsigned* flagp = (unsigned*)(myws + WS_FLAGS + (size_t)src * 256);
  run_tile<4, 1>(smem, myws + xoff + (size_t)mblk * BM * (GK * 2), a.w + (size_t)nblk * BN * 2, a.out + ((size_t)mblk * BM * NL + nblk * BN) * 2,
                 a.x + (size_t)mg * 32 * (GK * 2), a.out + ((size_t)(rank * MS + mg * 32) * NL + nblk * BN) * 2,
                 true, flagp, tmop, call, 0u, wid, lane TSARGS);
}

extern "C" void kernel_launch(int rank, int nranks, ncclComm_t comm,
                              hipStream_t stream,
                              void* const* d_in, const int* in_sizes, int n_in,
                              void* d_out, int out_size,
                              const int* in_dtypes, int out_dtype,
                              void* const* d_out_all,
                              void* const* d_ws_all, size_t ws_bytes) {
  if (nranks != NR || ws_bytes < WS_NEED || (unsigned)rank >= (unsigned)NR) { fprintf(stderr, "unexpected configuration\n"); return; }
  static bool init = false;
  if (!init) {
    HIP_CHECK(hipFuncSetAttribute((const void*)ag_gemm_kernel, hipFuncAttributeMaxDynamicSharedMemorySize, LDS_BYTES));
    init = true;
  }
  Args a;
  a.x = (const char*)d_in[0]; a.w = (const char*)d_in[1]; a.out = (char*)d_out; a.rank = rank;
  for (int p = 0; p < NR; ++p) a.ws[p] = (char*)d_ws_all[p];
  ag_gemm_kernel<<<NWG, 512, LDS_BYTES, stream>>>(a);
  HIP_CHECK(hipGetLastError());
}
```

```cpp
#include <hip/hip_runtime.h>
#include <rccl/rccl.h>
#include <cstdio>
#include <cstdint>

#define HIP_CHECK(expr) do { hipError_t _e = (expr); if (_e != hipSuccess) { \
    fprintf(stderr, "HIP error %s at %s:%d\n", hipGetErrorString(_e), __FILE__, __LINE__); \
    return; } } while (0)

typedef __attribute__((ext_vector_type(8))) short bf16x8;
typedef __attribute__((ext_vector_type(4))) short s16x4;
typedef __attribute__((ext_vector_type(4))) float f32x4;
typedef __attribute__((ext_vector_type(4))) unsigned u32x4;
typedef __attribute__((ext_vector_type(2))) unsigned u32x2;

constexpr int NR = 8;
constexpr int GM = 4096, GK = 4096, NL = 2048, MS = 512;
constexpr int BM = 256, BN = 128, BK = 64, NT = GK / BK;
#define NCHUNK 32
constexpr int NCH = NCHUNK, KC = GK / NCH, IPC = KC / BK;
constexpr int NWG = 256, NCW = 16;
constexpr int NSTAGE = 3;
constexpr int A_BYTES = BM * BK * 2, B_BYTES = BK * BN * 2, ST_BYTES = A_BYTES + B_BYTES;
constexpr int CTRL_OFF = NSTAGE * (ST_BYTES + 4096);
constexpr int LDS_BYTES = CTRL_OFF + 64;

constexpr size_t XBUF_BYTES = (size_t)GM * GK * 2;
constexpr size_t WS_CTRL = 2 * XBUF_BYTES;
constexpr size_t WS_FLAGS = WS_CTRL;
constexpr size_t WS_TICK = WS_CTRL + 16384;
constexpr size_t WS_TMO = WS_CTRL + 24576;
constexpr size_t WS_HEAD = WS_CTRL + 24576 + 1024;
#define INFL 7
constexpr size_t WS_CALL = WS_CTRL + 32768;
constexpr size_t WS_DBG = WS_CALL + (size_t)NWG * 128;
constexpr size_t WS_NEED = WS_DBG + (size_t)NWG * 64;
#define TS(k) do {} while (0)
constexpr unsigned long long DEADLINE_TICKS = 50000000ull;

struct Args {
  const char* x; const char* w; char* out; char* ws[NR]; int rank;
};

__device__ __forceinline__ int ldsA_byte(int r, int c) {
  int st = (r >> 4) * 2 + (c >> 5), ob = (r & 15) * 64 + (c & 31) * 2;
  return st * 1024 + (ob ^ (((ob >> 9) & 1) << 5));
}
__device__ __forceinline__ int swzB(int row) { return ((row & 3) << 2) | ((row >> 2) & 3); }

__device__ __forceinline__ float gelu_f(float y) {
  const float c2 = 2.0f * 0.7978845608028654f;
  float u = c2 * (y + 0.044715f * y * y * y);
  return y * __builtin_amdgcn_rcpf(1.0f + __expf(-u));
}
__device__ __forceinline__ unsigned pack_bf16(float a, float b) {
  typedef __attribute__((ext_vector_type(2))) __bf16 b2;
  b2 v; v[0] = (__bf16)a; v[1] = (__bf16)b;
  return __builtin_bit_cast(unsigned, v);
}

#define LDS_ADDR(p) ((unsigned)(size_t)(__attribute__((address_space(3))) char*)(p))

template <int MI, int EX>
__device__ __forceinline__ void run_tile(char* smem, const char* Abase, const char* Bbase, char* obase, const char* Xbase, char* xobase, bool gated,
                                         unsigned* flagp, unsigned* tmop, unsigned call, unsigned cnt0, int wid, int lane
                                         ) {
#define MSP 2
  constexpr int MSPLIT = MI == 1 ? 0 : MSP;
  constexpr int NLD = MI + 2 + EX;
  constexpr int AB = MI * 8192 + EX * 4096, STB = AB + B_BYTES, NS = CTRL_OFF / STB, D = NS - 1;
  unsigned aoff[MI], boff[2];
#pragma unroll
  for (int i = 0; i < MI; ++i) {
    int bb = (wid * MI + i) * 1024 + lane * 16;
    int st = bb >> 10, sb = bb & 1023, sw = sb ^ (((sb >> 9) & 1) << 5);
    int R = (st >> 1) * 16 + (sw >> 6), C = (st & 1) * 32 + ((sw & 63) >> 1);
    aoff[i] = (unsigned)(R * (GK * 2) + C * 2);
  }
#pragma unroll
  for (int i = 0; i < 2; ++i) {
    int bb = (wid * 2 + i) * 1024 + lane * 16;
    int row = bb >> 8, pos = (bb >> 4) & 15, ch = pos ^ swzB(row);
    boff[i] = (unsigned)(row * (NL * 2) + ch * 16);
  }
  unsigned xo = 0;
  if (EX) {
    int bb = wid * 512 + (lane & 31) * 16;
    int st = bb >> 10, sb = bb & 1023, sw = sb ^ (((sb >> 9) & 1) << 5);
    int R = (st >> 1) * 16 + (sw >> 6), C = (st & 1) * 32 + ((sw & 63) >> 1);
    xo = (unsigned)(R * (GK * 2) + C * 2);
  }
  auto stage = [&](int t, int s) {
    char* la = smem + s * STB + wid * (MI * 1024);
    char* lb = smem + s * STB + AB + wid * 2048;
    const char* ga = Abase + (size_t)t * (BK * 2);
    const char* gb = Bbase + (size_t)t * BK * (NL * 2);
#pragma unroll
    for (int i = 0; i < MI; ++i)
      __builtin_amdgcn_global_load_lds((const unsigned*)(ga + aoff[i]), (unsigned*)(la + i * 1024 + lane * 16), 16, 0, 0);
#pragma unroll
    for (int i = 0; i < 2; ++i)
      __builtin_amdgcn_global_load_lds((const unsigned*)(gb + boff[i]), (unsigned*)(lb + i * 1024 + lane * 16), 16, 0, 0);
    if (EX && lane < 32)
      __builtin_amdgcn_global_load_lds((const unsigned*)(Xbase + (size_t)t * (BK * 2) + xo), (unsigned*)(smem + s * STB + MI * 8192 + wid * 512 + lane * 16), 16, 0, 0);
  };
  const unsigned lds0 = LDS_ADDR(smem);
  const unsigned cnt_addr = lds0 + CTRL_OFF;
  unsigned rdy = 0;
  auto wait_chunk = [&](int c) {
    if (rdy > (unsigned)c) return;
    if (wid != 0) {
      for (;;) {
        unsigned r;
        asm volatile("ds_read_b32 %0, %1 offset:4\n\ts_waitcnt lgkmcnt(0)" : "=v"(r) : "v"(cnt_addr) : "memory");
        rdy = __builtin_amdgcn_readfirstlane(r);
        if (rdy > (unsigned)c) break;
        __builtin_amdgcn_s_sleep(1);
      }
      return;
    }
    unsigned long long t0 = __builtin_amdgcn_s_memrealtime();
    unsigned nr;
    for (;;) {
      unsigned f = __hip_atomic_load(flagp + (lane & (NCH - 1)), __ATOMIC_RELAXED, __HIP_MEMORY_SCOPE_SYSTEM);
      unsigned long long m = ~__builtin_amdgcn_ballot_w64(f >= call);
      nr = m ? (unsigned)__builtin_ctzll(m) : 64u;
      if (nr > (unsigned)c) break;
      if (__hip_atomic_load(tmop, __ATOMIC_RELAXED, __HIP_MEMORY_SCOPE_AGENT) == call ||
          __builtin_amdgcn_s_memrealtime() - t0 > DEADLINE_TICKS) {
        __hip_atomic_store(tmop, call, __ATOMIC_RELAXED, __HIP_MEMORY_SCOPE_AGENT);
        nr = 64u;
        break;
      }
      __builtin_amdgcn_s_sleep(2);
    }
    if (lane == 0) asm volatile("ds_write_b32 %0, %1 offset:4" :: "v"(cnt_addr), "v"(nr) : "memory");
    rdy = nr;
    if (c == 0) TS(2);
    if (c <= NCH / 2) TS(5);
    TS(6);
  };

  const int wm = wid >> 1, wn = wid & 1, fr = lane & 15, fq = lane >> 4;
  const int q = fr >> 2, p = lane & 3;
  const unsigned abase = lds0 + (unsigned)ldsA_byte(wm * (16 * MI) + fr, fq * 8);
  const unsigned ebase = lds0 + (unsigned)ldsA_byte(64 * MI + (wm >> 1) * 16 + fr, fq * 8);
  f32x4 eacc[2] = {(f32x4){0.f, 0.f, 0.f, 0.f}, (f32x4){0.f, 0.f, 0.f, 0.f}};
  unsigned bbase[4][2];
#pragma unroll
  for (int ni = 0; ni < 4; ++ni)
#pragma unroll
    for (int tt = 0; tt < 2; ++tt) {
      int row = 8 * fq + 4 * tt + q, ch = 2 * (wn * 4 + ni) + (p >> 1);
      bbase[ni][tt] = lds0 + AB + (unsigned)(256 * row + 16 * (ch ^ swzB(row)) + 8 * (p & 1));
    }
  f32x4 acc[MI][4];
#pragma unroll
  for (int i = 0; i < MI; ++i)
#pragma unroll
    for (int j = 0; j < 4; ++j) acc[i][j] = (f32x4){0.f, 0.f, 0.f, 0.f};

  bf16x8 af[2][MI], ef[2]; s16x4 bh[2][4][2];
  auto read_frags = [&](int kk, unsigned so) {
#pragma unroll
    for (int ni = 0; ni < 4; ++ni)
#pragma unroll
      for (int tt = 0; tt < 2; ++tt) {
        unsigned ba = bbase[ni][tt] + so;
        if (kk == 0) asm volatile("ds_read_b64_tr_b16 %0, %1" : "=v"(bh[kk][ni][tt]) : "v"(ba) : "memory");
        else asm volatile("ds_read_b64_tr_b16 %0, %1 offset:8192" : "=v"(bh[kk][ni][tt]) : "v"(ba) : "memory");
      }
#pragma unroll
    for (int mi = 0; mi < MI; ++mi)
      asm volatile("ds_read_b128 %0, %1 offset:%2" : "=v"(af[kk][mi]) : "v"(abase + so), "i"(mi * 2048 + kk * 1024) : "memory");
    if (EX) asm volatile("ds_read_b128 %0, %1 offset:%2" : "=v"(ef[kk]) : "v"(ebase + so), "i"(kk * 1024) : "memory");
  };
  auto pin = [&](int kk) {
#pragma unroll
    for (int mi = 0; mi < MI; ++mi) asm volatile("" : "+v"(af[kk][mi]));
#pragma unroll
    for (int ni = 0; ni < 4; ++ni) asm volatile("" : "+v"(bh[kk][ni][0]), "+v"(bh[kk][ni][1]));
    if (EX) asm volatile("" : "+v"(ef[kk]));
  };
  auto mma = [&](int kk, int m0, int m1) {
#pragma unroll
    for (int mi = m0; mi < m1; ++mi)
#pragma unroll
      for (int ni = 0; ni < 4; ++ni)
        acc[mi][ni] = __builtin_amdgcn_mfma_f32_16x16x32_bf16(__builtin_shufflevector(bh[kk][ni][0], bh[kk][ni][1], 0, 1, 2, 3, 4, 5, 6, 7), af[kk][mi], acc[mi][ni], 0, 0, 0);
    if (EX && m1 == MI) {
      if (wm & 1) {
        eacc[0] = __builtin_amdgcn_mfma_f32_16x16x32_bf16(__builtin_shufflevector(bh[kk][2][0], bh[kk][2][1], 0, 1, 2, 3, 4, 5, 6, 7), ef[kk], eacc[0], 0, 0, 0);
        eacc[1] = __builtin_amdgcn_mfma_f32_16x16x32_bf16(__builtin_shufflevector(bh[kk][3][0], bh[kk][3][1], 0, 1, 2, 3, 4, 5, 6, 7), ef[kk], eacc[1], 0, 0, 0);
      } else {
        eacc[0] = __builtin_amdgcn_mfma_f32_16x16x32_bf16(__builtin_shufflevector(bh[kk][0][0], bh[kk][0][1], 0, 1, 2, 3, 4, 5, 6, 7), ef[kk], eacc[0], 0, 0, 0);
        eacc[1] = __builtin_amdgcn_mfma_f32_16x16x32_bf16(__builtin_shufflevector(bh[kk][1][0], bh[kk][1][1], 0, 1, 2, 3, 4, 5, 6, 7), ef[kk], eacc[1], 0, 0, 0);
      }
    }
    __builtin_amdgcn_sched_barrier(0);
  };
  auto arrive = [&]() { if (lane == 0) asm volatile("ds_add_u32 %0, %1" :: "v"(cnt_addr), "v"(1u) : "memory"); };
  auto sync_stage = [&](int t) {
    const unsigned target = cnt0 + 8u * (unsigned)(t + 1);
    for (;;) {
      unsigned v;
      asm volatile("ds_read_b32 %0, %1\n\ts_waitcnt lgkmcnt(0)" : "=v"(v) : "v"(cnt_addr) : "memory");
      if (__builtin_amdgcn_readfirstlane(v) >= target) break;
    }
  };

#pragma unroll
  for (int i = 0; i < D; ++i) {
    if (gated && (i % IPC) == 0) wait_chunk(i / IPC);
    stage(i, i);
  }
  asm volatile("s_waitcnt vmcnt(%0)" :: "i"(NLD * (D - 1)) : "memory");
  arrive();
  int s = 0;
  for (int t = 0; t < NT; ++t) {
    sync_stage(t);
    if (t + D < NT) {
      if (gated && ((t + D) % IPC) == 0) wait_chunk((t + D) / IPC);
      int s2 = s + D; if (s2 >= NS) s2 -= NS;
      stage(t + D, s2);
    }
    read_frags(0, (unsigned)(s * STB));
    read_frags(1, (unsigned)(s * STB));
    asm volatile("s_waitcnt lgkmcnt(%0)" :: "i"(8 + MI + EX) : "memory");
    pin(0);
    mma(0, 0, MI);
    asm volatile("s_waitcnt lgkmcnt(0)" ::: "memory");
    pin(1);
    if (t + D < NT) asm volatile("s_waitcnt vmcnt(%0)" :: "i"(NLD * (D - 1)) : "memory");
    else if (D > 2 && t + D == NT) asm volatile("s_waitcnt vmcnt(%0)" :: "i"(NLD * (D > 2 ? D - 2 : 0)) : "memory");
    else if (D > 3 && t + D == NT + 1) asm volatile("s_waitcnt vmcnt(%0)" :: "i"(NLD * (D > 3 ? D - 3 : 0)) : "memory");
    else if (D > 4 && t + D == NT + 2) asm volatile("s_waitcnt vmcnt(%0)" :: "i"(NLD * (D > 4 ? D - 4 : 0)) : "memory");
    else asm volatile("s_waitcnt vmcnt(0)" ::: "memory");
    arrive();
    mma(1, 0, MI);
    s = s + 1; if (s >= NS) s = 0;
  }
  if (wid == 0 && gated) TS(3);

  const bool bad = gated && (__hip_atomic_load(tmop, __ATOMIC_RELAXED, __HIP_MEMORY_SCOPE_AGENT) == call);
  auto store_pair = [&](char* p, const f32x4& u, const f32x4& v, bool nan) {
    unsigned a0 = pack_bf16(gelu_f(u[0]), gelu_f(u[1])), a1 = pack_bf16(gelu_f(u[2]), gelu_f(u[3]));
    unsigned b0 = pack_bf16(gelu_f(v[0]), gelu_f(v[1])), b1 = pack_bf16(gelu_f(v[2]), gelu_f(v[3]));
    auto r0 = __builtin_amdgcn_permlane16_swap(a0, b0, false, false);
    auto r1 = __builtin_amdgcn_permlane16_swap(a1, b1, false, false);
    u32x4 o = {r0[0], r1[0], r0[1], r1[1]};
    if (nan) o = (u32x4){0x7FC07FC0u, 0x7FC07FC0u, 0x7FC07FC0u, 0x7FC07FC0u};
    *(u32x4*)p = o;
  };
  const int lcol = (fq & 1) * 16 + (fq & 2) * 4;
  char* ob = obase + ((size_t)(wm * (16 * MI) + fr) * NL + wn * 64 + lcol) * 2;
#pragma unroll
  for (int mi = 0; mi < MI; ++mi)
#pragma unroll
    for (int h = 0; h < 2; ++h) store_pair(ob + ((size_t)mi * 16 * NL + h * 32) * 2, acc[mi][2 * h], acc[mi][2 * h + 1], bad);
  if (EX) store_pair(xobase + ((size_t)((wm >> 1) * 16 + fr) * NL + wn * 64 + (wm & 1) * 32 + lcol) * 2, eacc[0], eacc[1], false);
}

__global__ __launch_bounds__(512, 1) void ag_gemm_kernel(Args a) {
  extern __shared__ __attribute__((aligned(16))) char smem[];
  const int tid = threadIdx.x, lane = tid & 63;
  const int wid = __builtin_amdgcn_readfirstlane(tid >> 6);
  const int b = blockIdx.x, rank = a.rank;
  char* myws = a.ws[rank];
  const int xcd = b & 7, idx = b >> 3;
  const bool is_comm = idx < NCW / 8;
  constexpr unsigned RPI = 512 / KC, LPR = KC / 8;
  constexpr unsigned IPCH = MS / RPI, NITEMS = NCH * IPCH;
  const unsigned lo = (unsigned)(lane / LPR) * (GK * 2) + (unsigned)(lane % LPR) * 16;
  unsigned it = (unsigned)((idx * 8 + xcd) * 8 + wid);
  unsigned off = (it & (IPCH - 1)) * (RPI * GK * 2) + (it / IPCH) * (KC * 2) + lo;
  u32x4 v0;
  if (is_comm) asm volatile("global_load_dwordx4 %0, %1, off" : "=v"(v0) : "v"(a.x + off) : "memory");
  volatile unsigned* ctl = (volatile unsigned*)(smem + CTRL_OFF);
  if (tid == 0) {
    unsigned* cp = (unsigned*)(myws + WS_CALL + (size_t)b * 128);
    unsigned c = __hip_atomic_load(cp, __ATOMIC_RELAXED, __HIP_MEMORY_SCOPE_AGENT) + 1u;
    __hip_atomic_store(cp, c, __ATOMIC_RELAXED, __HIP_MEMORY_SCOPE_AGENT);
    ctl[0] = 0; ctl[1] = 0; ctl[2] = c;
    if (!is_comm) {
      __builtin_amdgcn_fence(__ATOMIC_ACQUIRE, "");
      asm volatile("s_waitcnt vmcnt(0)" ::: "memory");
    }
  }
  __syncthreads();
  const unsigned call = __builtin_amdgcn_readfirstlane(ctl[2]);
  const size_t xoff = (call & 1u) ? XBUF_BYTES : 0;
  if (wid == 0) TS(0);

  if (is_comm) {
    const unsigned base = (call - 1u) * NITEMS - NCW * 8;
    char* headp = myws + WS_HEAD;
    const size_t dbase = xoff + (size_t)rank * MS * (GK * 2);
    char* pw = a.ws[0];
#pragma unroll
    for (int p = 1; p < NR; ++p) pw = (lane == p) ? a.ws[p] : pw;
    pw += WS_FLAGS + (size_t)rank * 256;
    char* dst[NR - 1];
#pragma unroll
    for (int j = 0; j < NR - 1; ++j) dst[j] = a.ws[(rank + 1 + (j + b * 8 + wid) % 7) & 7] + dbase;
    unsigned g = 0, tk = 0, pc = 0xffffffffu;
    while (it < NITEMS) {
      if (lane == 0) asm volatile("global_atomic_add %0, %1, %2, off sc0" : "=v"(g) : "v"(headp), "v"(1u) : "memory");
      asm volatile("s_waitcnt vmcnt(1)" : "+v"(v0), "+v"(tk) :: "memory");
      asm volatile("global_store_dwordx4 %0, %1, off sc0 sc1" :: "v"(dst[0] + off), "v"(v0) : "memory");
      if (pc != 0xffffffffu && (__builtin_amdgcn_readfirstlane(tk) & (IPCH - 1)) == IPCH - 1 && lane < NR && lane != rank)
        asm volatile("global_store_dword %0, %1, off sc0 sc1" :: "v"(pw + (size_t)pc * 4), "v"(call) : "memory");
#pragma unroll
      for (int j = 1; j < NR - 1; ++j) {
        asm volatile("s_waitcnt vmcnt(%0)" :: "i"(INFL - 1) : "memory");
        asm volatile("global_store_dwordx4 %0, %1, off sc0 sc1" :: "v"(dst[j] + off), "v"(v0) : "memory");
      }
      asm volatile("s_waitcnt vmcnt(6)" : "+v"(g) :: "memory");
      pc = it / IPCH;
      it = __builtin_amdgcn_readfirstlane(g) - base;
      off = (it & (IPCH - 1)) * (RPI * GK * 2) + (it / IPCH) * (KC * 2) + lo;
      asm volatile("s_waitcnt vmcnt(0)" ::: "memory");
      if (lane == 0) asm volatile("global_atomic_add %0, %1, %2, off sc0" : "=v"(tk) : "v"(myws + WS_TICK + (size_t)pc * 128), "v"(1u) : "memory");
      if (it < NITEMS) asm volatile("global_load_dwordx4 %0, %1, off" : "=v"(v0) : "v"(a.x + off) : "memory");
    }
    asm volatile("s_waitcnt vmcnt(0)" : "+v"(tk) :: "memory");
    if (pc != 0xffffffffu && (__builtin_amdgcn_readfirstlane(tk) & (IPCH - 1)) == IPCH - 1 && lane < NR && lane != rank)
      asm volatile("global_store_dword %0, %1, off sc0 sc1" :: "v"(pw + (size_t)pc * 4), "v"(call) : "memory");
    if (wid == 0) TS(1);
    return;
  }

  unsigned* tmop = (unsigned*)(myws + WS_TMO);
#define TSARGS
  if (idx < NCW / 8 + 2) {
    const int nblk = xcd * 2 + (idx - NCW / 8);
    run_tile<1, 0>(smem, a.x + (size_t)448 * (GK * 2), a.w + (size_t)nblk * BN * 2, a.out + ((size_t)(rank * MS + 448) * NL + nblk * BN) * 2,
                   nullptr, nullptr, false, nullptr, tmop, call, 0u, wid, lane TSARGS);
    if (wid == 0) TS(7);
    return;
  }
  const int gi = idx - (NCW / 8 + 2);
  const int mi7 = gi % 7, nblk = (xcd & 3) * 4 + gi / 7;
  const int mg = (xcd >> 2) * 7 + mi7;
  const int mblk = (2 * (rank + 1) + mg) & 15;
  const int src = mblk >> 1;
  unsigned* flagp = (unsigned*)(myws + WS_FLAGS + (size_t)src * 256);
  run_tile<4, 1>(smem, myws + xoff + (size_t)mblk * BM * (GK * 2), a.w + (size_t)nblk * BN * 2, a.out + ((size_t)mblk * BM * NL + nblk * BN) * 2,
                 a.x + (size_t)mg * 32 * (GK * 2), a.out + ((size_t)(rank * MS + mg * 32) * NL + nblk * BN) * 2,
                 true, flagp, tmop, call, 0u, wid, lane TSARGS);
}

extern "C" void kernel_launch(int rank, int nranks, ncclComm_t comm,
                              hipStream_t stream,
                              void* const* d_in, const int* in_sizes, int n_in,
                              void* d_out, int out_size,
                              const int* in_dtypes, int out_dtype,
                              void* const* d_out_all,
                              void* const* d_ws_all, size_t ws_bytes) {
  if (nranks != NR || ws_bytes < WS_NEED || (unsigned)rank >= (unsigned)NR) { fprintf(stderr, "unexpected configuration\n"); return; }
  static bool init = false;
  if (!init) {
    HIP_CHECK(hipFuncSetAttribute((const void*)ag_gemm_kernel, hipFuncAttributeMaxDynamicSharedMemorySize, LDS_BYTES));
    init = true;
  }
  Args a;
  a.x = (const char*)d_in[0]; a.w = (const char*)d_in[1]; a.out = (char*)d_out; a.rank = rank;
  for (int p = 0; p < NR; ++p) a.ws[p] = (char*)d_ws_all[p];
  ag_gemm_kernel<<<NWG, 512, LDS_BYTES, stream>>>(a);
  HIP_CHECK(hipGetLastError());
}
```

```cpp
#include <hip/hip_runtime.h>
#include <rccl/rccl.h>
#include <cstdio>
#include <cstdint>

#define HIP_CHECK(expr) do { hipError_t _e = (expr); if (_e != hipSuccess) { \
    fprintf(stderr, "HIP error %s at %s:%d\n", hipGetErrorString(_e), __FILE__, __LINE__); \
    return; } } while (0)

typedef __attribute__((ext_vector_type(8))) short bf16x8;
typedef __attribute__((ext_vector_type(4))) short s16x4;
typedef __attribute__((ext_vector_type(4))) float f32x4;
typedef __attribute__((ext_vector_type(4))) unsigned u32x4;
typedef __attribute__((ext_vector_type(2))) unsigned u32x2;

constexpr int NR = 8;
constexpr int GM = 4096, GK = 4096, NL = 2048, MS = 512;
constexpr int BM = 256, BN = 128, BK = 64, NT = GK / BK;
#define NCHUNK 32
constexpr int NCH = NCHUNK, KC = GK / NCH, IPC = KC / BK;
constexpr int NWG = 256, NCW = 16;
constexpr int NSTAGE = 3;
constexpr int A_BYTES = BM * BK * 2, B_BYTES = BK * BN * 2, ST_BYTES = A_BYTES + B_BYTES;
constexpr int CTRL_OFF = NSTAGE * (ST_BYTES + 4096);
constexpr int LDS_BYTES = CTRL_OFF + 64;

constexpr size_t XBUF_BYTES = (size_t)GM * GK * 2;
constexpr size_t WS_CTRL = 2 * XBUF_BYTES;
constexpr size_t WS_FLAGS = WS_CTRL;
constexpr size_t WS_TICK = WS_CTRL + 16384;
constexpr size_t WS_TMO = WS_CTRL + 24576;
constexpr size_t WS_HEAD = WS_CTRL + 24576 + 1024;
#define INFL 7
constexpr size_t WS_CALL = WS_CTRL + 32768;
constexpr size_t WS_DBG = WS_CALL + (size_t)NWG * 128;
constexpr size_t WS_NEED = WS_DBG + (size_t)NWG * 64;
#define TS(k) do {} while (0)
constexpr unsigned long long DEADLINE_TICKS = 50000000ull;

struct Args {
  const char* x; const char* w; char* out; char* ws[NR]; int rank;
};

__device__ __forceinline__ int ldsA_byte(int r, int c) {
  int st = (r >> 4) * 2 + (c >> 5), ob = (r & 15) * 64 + (c & 31) * 2;
  return st * 1024 + (ob ^ (((ob >> 9) & 1) << 5));
}
__device__ __forceinline__ int swzB(int row) { return ((row & 3) << 2) | ((row >> 2) & 3); }

__device__ __forceinline__ float gelu_f(float y) {
  const float c2 = 2.0f * 0.7978845608028654f;
  float u = c2 * (y + 0.044715f * y * y * y);
  return y * __builtin_amdgcn_rcpf(1.0f + __expf(-u));
}
__device__ __forceinline__ unsigned pack_bf16(float a, float b) {
  typedef __attribute__((ext_vector_type(2))) __bf16 b2;
  b2 v; v[0] = (__bf16)a; v[1] = (__bf16)b;
  return __builtin_bit_cast(unsigned, v);
}

#define LDS_ADDR(p) ((unsigned)(size_t)(__attribute__((address_space(3))) char*)(p))

template <int MI, int EX>
__device__ __forceinline__ void run_tile(char* smem, const char* Abase, const char* Bbase, char* obase, const char* Xbase, char* xobase, bool gated,
                                         unsigned* flagp, unsigned* tmop, unsigned call, unsigned cnt0, int wid, int lane
                                         ) {
#define MSP 2
  constexpr int MSPLIT = MI == 1 ? 0 : MSP;
  constexpr int NLD = MI + 2 + EX;
  constexpr int AB = MI * 8192 + EX * 4096, STB = AB + B_BYTES, NS = CTRL_OFF / STB, D = NS - 1;
  unsigned aoff[MI], boff[2];
#pragma unroll
  for (int i = 0; i < MI; ++i) {
    int bb = (wid * MI + i) * 1024 + lane * 16;
    int st = bb >> 10, sb = bb & 1023, sw = sb ^ (((sb >> 9) & 1) << 5);
    int R = (st >> 1) * 16 + (sw >> 6), C = (st & 1) * 32 + ((sw & 63) >> 1);
    aoff[i] = (unsigned)(R * (GK * 2) + C * 2);
  }
#pragma unroll
  for (int i = 0; i < 2; ++i) {
    int bb = (wid * 2 + i) * 1024 + lane * 16;
    int row = bb >> 8, pos = (bb >> 4) & 15, ch = pos ^ swzB(row);
    boff[i] = (unsigned)(row * (NL * 2) + ch * 16);
  }
  unsigned xo = 0;
  if (EX) {
    int bb = wid * 512 + (lane & 31) * 16;
    int st = bb >> 10, sb = bb & 1023, sw = sb ^ (((sb >> 9) & 1) << 5);
    int R = (st >> 1) * 16 + (sw >> 6), C = (st & 1) * 32 + ((sw & 63) >> 1);
    xo = (unsigned)(R * (GK * 2) + C * 2);
  }
  auto stage_part = [&](int t, int s, int j) {
    if (j < MI)
      __builtin_amdgcn_global_load_lds((const unsigned*)(Abase + (size_t)t * (BK * 2) + aoff[j < MI ? j : 0]),
                                       (unsigned*)(smem + s * STB + (wid * MI + j) * 1024 + lane * 16), 16, 0, 0);
    else if (j < MI + 2)
      __builtin_amdgcn_global_load_lds((const unsigned*)(Bbase + (size_t)t * BK * (NL * 2) + boff[j == MI ? 0 : 1]),
                                       (unsigned*)(smem + s * STB + AB + (wid * 2 + j - MI) * 1024 + lane * 16), 16, 0, 0);
    else if (EX && lane < 32)
      __builtin_amdgcn_global_load_lds((const unsigned*)(Xbase + (size_t)t * (BK * 2) + xo), (unsigned*)(smem + s * STB + MI * 8192 + wid * 512 + lane * 16), 16, 0, 0);
  };
  auto stage = [&](int t, int s) {
#pragma unroll
    for (int j = 0; j < NLD; ++j) stage_part(t, s, j);
  };
  const unsigned lds0 = LDS_ADDR(smem);
  const unsigned cnt_addr = lds0 + CTRL_OFF;
  unsigned rdy = 0;
  auto wait_chunk = [&](int c) {
    if (rdy > (unsigned)c) return;
    if (wid != 0) {
      for (;;) {
        unsigned r;
        asm volatile("ds_read_b32 %0, %1 offset:4\n\ts_waitcnt lgkmcnt(0)" : "=v"(r) : "v"(cnt_addr) : "memory");
        rdy = __builtin_amdgcn_readfirstlane(r);
        if (rdy > (unsigned)c) break;
        __builtin_amdgcn_s_sleep(1);
      }
      return;
    }
    unsigned long long t0 = __builtin_amdgcn_s_memrealtime();
    unsigned nr;
    for (;;) {
      unsigned f = __hip_atomic_load(flagp + (lane & (NCH - 1)), __ATOMIC_RELAXED, __HIP_MEMORY_SCOPE_SYSTEM);
      unsigned long long m = ~__builtin_amdgcn_ballot_w64(f >= call);
      nr = m ? (unsigned)__builtin_ctzll(m) : 64u;
      if (nr > (unsigned)c) break;
      if (__hip_atomic_load(tmop, __ATOMIC_RELAXED, __HIP_MEMORY_SCOPE_AGENT) == call ||
          __builtin_amdgcn_s_memrealtime() - t0 > DEADLINE_TICKS) {
        __hip_atomic_store(tmop, call, __ATOMIC_RELAXED, __HIP_MEMORY_SCOPE_AGENT);
        nr = 64u;
        break;
      }
      __builtin_amdgcn_s_sleep(2);
    }
    if (lane == 0) asm volatile("ds_write_b32 %0, %1 offset:4" :: "v"(cnt_addr), "v"(nr) : "memory");
    rdy = nr;
    if (c == 0) TS(2);
    if (c <= NCH / 2) TS(5);
    TS(6);
  };

  const int wm = wid >> 1, wn = wid & 1, fr = lane & 15, fq = lane >> 4;
  const int q = fr >> 2, p = lane & 3;
  const unsigned abase = lds0 + (unsigned)ldsA_byte(wm * (16 * MI) + fr, fq * 8);
  const unsigned ebase = lds0 + (unsigned)ldsA_byte(64 * MI + (wm >> 1) * 16 + fr, fq * 8);
  f32x4 eacc[2] = {(f32x4){0.f, 0.f, 0.f, 0.f}, (f32x4){0.f, 0.f, 0.f, 0.f}};
  unsigned bbase[4][2];
#pragma unroll
  for (int ni = 0; ni < 4; ++ni)
#pragma unroll
    for (int tt = 0; tt < 2; ++tt) {
      int row = 8 * fq + 4 * tt + q, ch = 2 * (wn * 4 + ni) + (p >> 1);
      bbase[ni][tt] = lds0 + AB + (unsigned)(256 * row + 16 * (ch ^ swzB(row)) + 8 * (p & 1));
    }
  f32x4 acc[MI][4];
#pragma unroll
  for (int i = 0; i < MI; ++i)
#pragma unroll
    for (int j = 0; j < 4; ++j) acc[i][j] = (f32x4){0.f, 0.f, 0.f, 0.f};

  bf16x8 af[2][MI], ef[2]; s16x4 bh[2][4][2];
  constexpr int NRD = 8 + MI + EX;
  auto read_part = [&](int kk, unsigned so, int j) {
    if (j < 8) {
      const int ni = j >> 1, tt = j & 1;
      unsigned ba = bbase[ni][tt] + so;
      if (kk == 0) asm volatile("ds_read_b64_tr_b16 %0, %1" : "=v"(bh[kk][ni][tt]) : "v"(ba) : "memory");
      else asm volatile("ds_read_b64_tr_b16 %0, %1 offset:8192" : "=v"(bh[kk][ni][tt]) : "v"(ba) : "memory");
    } else if (j < 8 + MI) {
      const int mi = j < 8 + MI ? j - 8 : 0;
      asm volatile("ds_read_b128 %0, %1 offset:%2" : "=v"(af[kk][mi]) : "v"(abase + so), "i"(mi * 2048 + kk * 1024) : "memory");
    } else if (EX) asm volatile("ds_read_b128 %0, %1 offset:%2" : "=v"(ef[kk]) : "v"(ebase + so), "i"(kk * 1024) : "memory");
  };
  auto read_frags = [&](int kk, unsigned so) {
#pragma unroll
    for (int j = 0; j < NRD; ++j) read_part(kk, so, j);
  };
  auto pin = [&](int kk) {
#pragma unroll
    for (int mi = 0; mi < MI; ++mi) asm volatile("" : "+v"(af[kk][mi]));
#pragma unroll
    for (int ni = 0; ni < 4; ++ni) asm volatile("" : "+v"(bh[kk][ni][0]), "+v"(bh[kk][ni][1]));
    if (EX) asm volatile("" : "+v"(ef[kk]));
  };
  auto mma = [&](int kk, bool ld, int tl, int sl, int j0, int j1, bool rd = false, unsigned rso = 0) {
    constexpr int NM = MI * 4;
#pragma unroll
    for (int i = 0; i < NM; ++i) {
      const int mi = i >> 2, ni = i & 3;
      acc[mi][ni] = __builtin_amdgcn_mfma_f32_16x16x32_bf16(__builtin_shufflevector(bh[kk][ni][0], bh[kk][ni][1], 0, 1, 2, 3, 4, 5, 6, 7), af[kk][mi], acc[mi][ni], 0, 0, 0);
      if (ld) {
#pragma unroll
        for (int j = j0; j < j1; ++j)
          if (((j - j0) * NM) / (j1 - j0) == i) { __builtin_amdgcn_sched_barrier(0); stage_part(tl, sl, j); __builtin_amdgcn_sched_barrier(0); }
      }
    }
    if (EX) {
      if (wm & 1) {
        eacc[0] = __builtin_amdgcn_mfma_f32_16x16x32_bf16(__builtin_shufflevector(bh[kk][2][0], bh[kk][2][1], 0, 1, 2, 3, 4, 5, 6, 7), ef[kk], eacc[0], 0, 0, 0);
        eacc[1] = __builtin_amdgcn_mfma_f32_16x16x32_bf16(__builtin_shufflevector(bh[kk][3][0], bh[kk][3][1], 0, 1, 2, 3, 4, 5, 6, 7), ef[kk], eacc[1], 0, 0, 0);
      } else {
        eacc[0] = __builtin_amdgcn_mfma_f32_16x16x32_bf16(__builtin_shufflevector(bh[kk][0][0], bh[kk][0][1], 0, 1, 2, 3, 4, 5, 6, 7), ef[kk], eacc[0], 0, 0, 0);
        eacc[1] = __builtin_amdgcn_mfma_f32_16x16x32_bf16(__builtin_shufflevector(bh[kk][1][0], bh[kk][1][1], 0, 1, 2, 3, 4, 5, 6, 7), ef[kk], eacc[1], 0, 0, 0);
      }
    }
    __builtin_amdgcn_sched_barrier(0);
  };
  auto arrive = [&]() { if (lane == 0) asm volatile("ds_add_u32 %0, %1" :: "v"(cnt_addr), "v"(1u) : "memory"); };
  auto sync_stage = [&](int t) {
    const unsigned target = cnt0 + 8u * (unsigned)(t + 1);
    for (;;) {
      unsigned v;
      asm volatile("ds_read_b32 %0, %1\n\ts_waitcnt lgkmcnt(0)" : "=v"(v) : "v"(cnt_addr) : "memory");
      if (__builtin_amdgcn_readfirstlane(v) >= target) break;
    }
  };

#pragma unroll
  for (int i = 0; i < D; ++i) {
    if (gated && (i % IPC) == 0) wait_chunk(i / IPC);
    stage(i, i);
  }
  asm volatile("s_waitcnt vmcnt(%0)" :: "i"(NLD * (D - 1)) : "memory");
  arrive();
#define NL1 4
  constexpr int L1 = NL1 < NLD ? NL1 : 0, L0 = NLD - L1;
  int s = 0;
  for (int t = 0; t < NT - D; ++t) {
    sync_stage(t);
    const bool now = true;
    if (gated && ((t + D) % IPC) == 0) wait_chunk((t + D) / IPC);
    int s2 = s + D; if (s2 >= NS) s2 -= NS;
    read_frags(0, (unsigned)(s * STB));
    read_frags(1, (unsigned)(s * STB));
    asm volatile("s_waitcnt lgkmcnt(%0)" :: "i"(8 + MI + EX) : "memory");
    pin(0);
    mma(0, now, t + D, s2, 0, L0, true, (unsigned)(s * STB));
    asm volatile("s_waitcnt lgkmcnt(0)" ::: "memory");
    pin(1);
    if (now) asm volatile("s_waitcnt vmcnt(%0)" :: "i"(NLD * (D - 2) + L0) : "memory");
    else asm volatile("s_waitcnt vmcnt(%0)" :: "i"(NLD * (D - 2)) : "memory");
    arrive();
    mma(1, now && L1 > 0, t + D, s2, L0, NLD);
    if (!now) { wait_chunk((t + D) / IPC); stage(t + D, s2); }
    s = s + 1; if (s >= NS) s = 0;
  }
  for (int t = NT - D; t < NT; ++t) {
    sync_stage(t);
    read_frags(0, (unsigned)(s * STB));
    read_frags(1, (unsigned)(s * STB));
    asm volatile("s_waitcnt lgkmcnt(%0)" :: "i"(8 + MI + EX) : "memory");
    pin(0);
    mma(0, false, 0, 0, 0, 1);
    asm volatile("s_waitcnt lgkmcnt(0)" ::: "memory");
    pin(1);
    if (D > 2 && t + D == NT) asm volatile("s_waitcnt vmcnt(%0)" :: "i"(NLD * (D > 2 ? D - 2 : 0)) : "memory");
    else if (D > 3 && t + D == NT + 1) asm volatile("s_waitcnt vmcnt(%0)" :: "i"(NLD * (D > 3 ? D - 3 : 0)) : "memory");
    else if (D > 4 && t + D == NT + 2) asm volatile("s_waitcnt vmcnt(%0)" :: "i"(NLD * (D > 4 ? D - 4 : 0)) : "memory");
    else asm volatile("s_waitcnt vmcnt(0)" ::: "memory");
    arrive();
    mma(1, false, 0, 0, 0, 1);
    s = s + 1; if (s >= NS) s = 0;
  }
  if (wid == 0 && gated) TS(3);

  const bool bad = gated && (__hip_atomic_load(tmop, __ATOMIC_RELAXED, __HIP_MEMORY_SCOPE_AGENT) == call);
  auto store_pair = [&](char* p, const f32x4& u, const f32x4& v, bool nan) {
    unsigned a0 = pack_bf16(gelu_f(u[0]), gelu_f(u[1])), a1 = pack_bf16(gelu_f(u[2]), gelu_f(u[3]));
    unsigned b0 = pack_bf16(gelu_f(v[0]), gelu_f(v[1])), b1 = pack_bf16(gelu_f(v[2]), gelu_f(v[3]));
    auto r0 = __builtin_amdgcn_permlane16_swap(a0, b0, false, false);
    auto r1 = __builtin_amdgcn_permlane16_swap(a1, b1, false, false);
    u32x4 o = {r0[0], r1[0], r0[1], r1[1]};
    if (nan) o = (u32x4){0x7FC07FC0u, 0x7FC07FC0u, 0x7FC07FC0u, 0x7FC07FC0u};
    *(u32x4*)p = o;
  };
  const int lcol = (fq & 1) * 16 + (fq & 2) * 4;
  char* ob = obase + ((size_t)(wm * (16 * MI) + fr) * NL + wn * 64 + lcol) * 2;
#pragma unroll
  for (int mi = 0; mi < MI; ++mi)
#pragma unroll
    for (int h = 0; h < 2; ++h) store_pair(ob + ((size_t)mi * 16 * NL + h * 32) * 2, acc[mi][2 * h], acc[mi][2 * h + 1], bad);
  if (EX) store_pair(xobase + ((size_t)((wm >> 1) * 16 + fr) * NL + wn * 64 + (wm & 1) * 32 + lcol) * 2, eacc[0], eacc[1], false);
}

__global__ __launch_bounds__(512, 1) void ag_gemm_kernel(Args a) {
  extern __shared__ __attribute__((aligned(16))) char smem[];
  const int tid = threadIdx.x, lane = tid & 63;
  const int wid = __builtin_amdgcn_readfirstlane(tid >> 6);
  const int b = blockIdx.x, rank = a.rank;
  char* myws = a.ws[rank];
  const int xcd = b & 7, idx = b >> 3;
  const bool is_comm = idx < NCW / 8;
  constexpr unsigned RPI = 512 / KC, LPR = KC / 8;
  constexpr unsigned IPCH = MS / RPI, NITEMS = NCH * IPCH;
  const unsigned lo = (unsigned)(lane / LPR) * (GK * 2) + (unsigned)(lane % LPR) * 16;
  unsigned it = (unsigned)((idx * 8 + xcd) * 8 + wid);
  unsigned off = (it & (IPCH - 1)) * (RPI * GK * 2) + (it / IPCH) * (KC * 2) + lo;
  u32x4 v0;
  if (is_comm) asm volatile("global_load_dwordx4 %0, %1, off" : "=v"(v0) : "v"(a.x + off) : "memory");
  volatile unsigned* ctl = (volatile unsigned*)(smem + CTRL_OFF);
  if (tid == 0) {
    unsigned* cp = (unsigned*)(myws + WS_CALL + (size_t)b * 128);
    unsigned c = __hip_atomic_load(cp, __ATOMIC_RELAXED, __HIP_MEMORY_SCOPE_AGENT) + 1u;
    __hip_atomic_store(cp, c, __ATOMIC_RELAXED, __HIP_MEMORY_SCOPE_AGENT);
    ctl[0] = 0; ctl[1] = 0; ctl[2] = c;
    if (!is_comm) {
      __builtin_amdgcn_fence(__ATOMIC_ACQUIRE, "");
      asm volatile("s_waitcnt vmcnt(0)" ::: "memory");
    }
  }
  __syncthreads();
  const unsigned call = __builtin_amdgcn_readfirstlane(ctl[2]);
  const size_t xoff = (call & 1u) ? XBUF_BYTES : 0;
  if (wid == 0) TS(0);

  if (is_comm) {
    const unsigned base = (call - 1u) * NITEMS - NCW * 8;
    char* headp = myws + WS_HEAD;
    const size_t dbase = xoff + (size_t)rank * MS * (GK * 2);
    char* pw = a.ws[0];
#pragma unroll
    for (int p = 1; p < NR; ++p) pw = (lane == p) ? a.ws[p] : pw;
    pw += WS_FLAGS + (size_t)rank * 256;
    char* dst[NR - 1];
#pragma unroll
    for (int j = 0; j < NR - 1; ++j) dst[j] = a.ws[(rank + 1 + (j + b * 8 + wid) % 7) & 7] + dbase;
    unsigned g = 0, tk = 0, pc = 0xffffffffu;
    while (it < NITEMS) {
      if (lane == 0) asm volatile("global_atomic_add %0, %1, %2, off sc0" : "=v"(g) : "v"(headp), "v"(1u) : "memory");
      asm volatile("s_waitcnt vmcnt(1)" : "+v"(v0), "+v"(tk) :: "memory");
      asm volatile("global_store_dwordx4 %0, %1, off sc0 sc1" :: "v"(dst[0] + off), "v"(v0) : "memory");
      if (pc != 0xffffffffu && (__builtin_amdgcn_readfirstlane(tk) & (IPCH - 1)) == IPCH - 1 && lane < NR && lane != rank)
        asm volatile("global_store_dword %0, %1, off sc0 sc1" :: "v"(pw + (size_t)pc * 4), "v"(call) : "memory");
#pragma unroll
      for (int j = 1; j < NR - 1; ++j) {
        asm volatile("s_waitcnt vmcnt(%0)" :: "i"(INFL - 1) : "memory");
        asm volatile("global_store_dwordx4 %0, %1, off sc0 sc1" :: "v"(dst[j] + off), "v"(v0) : "memory");
      }
      asm volatile("s_waitcnt vmcnt(6)" : "+v"(g) :: "memory");
      pc = it / IPCH;
      it = __builtin_amdgcn_readfirstlane(g) - base;
      off = (it & (IPCH - 1)) * (RPI * GK * 2) + (it / IPCH) * (KC * 2) + lo;
      asm volatile("s_waitcnt vmcnt(0)" ::: "memory");
      if (lane == 0) asm volatile("global_atomic_add %0, %1, %2, off sc0" : "=v"(tk) : "v"(myws + WS_TICK + (size_t)pc * 128), "v"(1u) : "memory");
      if (it < NITEMS) asm volatile("global_load_dwordx4 %0, %1, off" : "=v"(v0) : "v"(a.x + off) : "memory");
    }
    asm volatile("s_waitcnt vmcnt(0)" : "+v"(tk) :: "memory");
    if (pc != 0xffffffffu && (__builtin_amdgcn_readfirstlane(tk) & (IPCH - 1)) == IPCH - 1 && lane < NR && lane != rank)
      asm volatile("global_store_dword %0, %1, off sc0 sc1" :: "v"(pw + (size_t)pc * 4), "v"(call) : "memory");
    if (wid == 0) TS(1);
    return;
  }

  unsigned* tmop = (unsigned*)(myws + WS_TMO);
#define TSARGS
  if (idx < NCW / 8 + 2) {
    const int nblk = xcd * 2 + (idx - NCW / 8);
    run_tile<1, 0>(smem, a.x + (size_t)448 * (GK * 2), a.w + (size_t)nblk * BN * 2, a.out + ((size_t)(rank * MS + 448) * NL + nblk * BN) * 2,
                   nullptr, nullptr, false, nullptr, tmop, call, 0u, wid, lane TSARGS);
    if (wid == 0) TS(7);
    return;
  }
  const int gi = idx - (NCW / 8 + 2);
  const int mi7 = gi % 7, nblk = (xcd & 3) * 4 + gi / 7;
  const int mg = (xcd >> 2) * 7 + mi7;
  const int mblk = (2 * (rank + 1) + mg) & 15;
  const int src = mblk >> 1;
  unsigned* flagp = (unsigned*)(myws + WS_FLAGS + (size_t)src * 256);
  run_tile<4, 1>(smem, myws + xoff + (size_t)mblk * BM * (GK * 2), a.w + (size_t)nblk * BN * 2, a.out + ((size_t)mblk * BM * NL + nblk * BN) * 2,
                 a.x + (size_t)mg * 32 * (GK * 2), a.out + ((size_t)(rank * MS + mg * 32) * NL + nblk * BN) * 2,
                 true, flagp, tmop, call, 0u, wid, lane TSARGS);
}

extern "C" void kernel_launch(int rank, int nranks, ncclComm_t comm,
                              hipStream_t stream,
                              void* const* d_in, const int* in_sizes, int n_in,
                              void* d_out, int out_size,
                              const int* in_dtypes, int out_dtype,
                              void* const* d_out_all,
                              void* const* d_ws_all, size_t ws_bytes) {
  if (nranks != NR || ws_bytes < WS_NEED || (unsigned)rank >= (unsigned)NR) { fprintf(stderr, "unexpected configuration\n"); return; }
  static bool init = false;
  if (!init) {
    HIP_CHECK(hipFuncSetAttribute((const void*)ag_gemm_kernel, hipFuncAttributeMaxDynamicSharedMemorySize, LDS_BYTES));
    init = true;
  }
  Args a;
  a.x = (const char*)d_in[0]; a.w = (const char*)d_in[1]; a.out = (char*)d_out; a.rank = rank;
  for (int p = 0; p < NR; ++p) a.ws[p] = (char*)d_ws_all[p];
  ag_gemm_kernel<<<NWG, 512, LDS_BYTES, stream>>>(a);
  HIP_CHECK(hipGetLastError());
}
```

```cpp
#include <hip/hip_runtime.h>
#include <rccl/rccl.h>
#include <cstdio>
#include <cstdint>

#define HIP_CHECK(expr) do { hipError_t _e = (expr); if (_e != hipSuccess) { \
    fprintf(stderr, "HIP error %s at %s:%d\n", hipGetErrorString(_e), __FILE__, __LINE__); \
    return; } } while (0)

typedef __attribute__((ext_vector_type(8))) short bf16x8;
typedef __attribute__((ext_vector_type(4))) short s16x4;
typedef __attribute__((ext_vector_type(4))) float f32x4;
typedef __attribute__((ext_vector_type(4))) unsigned u32x4;
typedef __attribute__((ext_vector_type(2))) unsigned u32x2;

constexpr int NR = 8;
constexpr int GM = 4096, GK = 4096, NL = 2048, MS = 512;
constexpr int BM = 256, BN = 128, BK = 64, NT = GK / BK;
#define NCHUNK 32
constexpr int NCH = NCHUNK, KC = GK / NCH, IPC = KC / BK;
constexpr int NWG = 256, NCW = 16;
constexpr int NSTAGE = 3;
constexpr int A_BYTES = BM * BK * 2, B_BYTES = BK * BN * 2, ST_BYTES = A_BYTES + B_BYTES;
constexpr int CTRL_OFF = NSTAGE * (ST_BYTES + 4096);
constexpr int LDS_BYTES = CTRL_OFF + 64;

constexpr size_t XBUF_BYTES = (size_t)GM * GK * 2;
constexpr size_t WS_CTRL = 2 * XBUF_BYTES;
constexpr size_t WS_FLAGS = WS_CTRL;
constexpr size_t WS_TICK = WS_CTRL + 16384;
constexpr size_t WS_TMO = WS_CTRL + 24576;
constexpr size_t WS_HEAD = WS_CTRL + 24576 + 1024;
#define INFL 7
constexpr size_t WS_CALL = WS_CTRL + 32768;
constexpr size_t WS_DBG = WS_CALL + (size_t)NWG * 128;
constexpr size_t WS_NEED = WS_DBG + (size_t)NWG * 64;
#define TS(k) do {} while (0)
constexpr unsigned long long DEADLINE_TICKS = 50000000ull;

struct Args {
  const char* x; const char* w; char* out; char* ws[NR]; int rank;
};

__device__ __forceinline__ int ldsA_byte(int r, int c) {
  int st = (r >> 4) * 2 + (c >> 5), ob = (r & 15) * 64 + (c & 31) * 2;
  return st * 1024 + (ob ^ (((ob >> 9) & 1) << 5));
}
__device__ __forceinline__ int swzB(int row) { return ((row & 3) << 2) | ((row >> 2) & 3); }

__device__ __forceinline__ float gelu_f(float y) {
  const float c2 = 2.0f * 0.7978845608028654f;
  float u = c2 * (y + 0.044715f * y * y * y);
  return y * __builtin_amdgcn_rcpf(1.0f + __expf(-u));
}
__device__ __forceinline__ unsigned pack_bf16(float a, float b) {
  typedef __attribute__((ext_vector_type(2))) __bf16 b2;
  b2 v; v[0] = (__bf16)a; v[1] = (__bf16)b;
  return __builtin_bit_cast(unsigned, v);
}

#define LDS_ADDR(p) ((unsigned)(size_t)(__attribute__((address_space(3))) char*)(p))

template <int MI, int EX>
__device__ __forceinline__ void run_tile(char* smem, const char* Abase, const char* Bbase, char* obase, const char* Xbase, char* xobase, bool gated,
                                         unsigned* flagp, unsigned* tmop, unsigned call, unsigned cnt0, int wid, int lane
                                         ) {
#define MSP 2
  constexpr int MSPLIT = MI == 1 ? 0 : MSP;
  constexpr int NLD = MI + 2 + EX;
  constexpr int AB = MI * 8192 + EX * 4096, STB = AB + B_BYTES, NS = CTRL_OFF / STB, D = NS - 1;
  unsigned aoff[MI], boff[2];
#pragma unroll
  for (int i = 0; i < MI; ++i) {
    int bb = (wid * MI + i) * 1024 + lane * 16;
    int st = bb >> 10, sb = bb & 1023, sw = sb ^ (((sb >> 9) & 1) << 5);
    int R = (st >> 1) * 16 + (sw >> 6), C = (st & 1) * 32 + ((sw & 63) >> 1);
    aoff[i] = (unsigned)(R * (GK * 2) + C * 2);
  }
#pragma unroll
  for (int i = 0; i < 2; ++i) {
    int bb = (wid * 2 + i) * 1024 + lane * 16;
    int row = bb >> 8, pos = (bb >> 4) & 15, ch = pos ^ swzB(row);
    boff[i] = (unsigned)(row * (NL * 2) + ch * 16);
  }
  unsigned xo = 0;
  if (EX) {
    int bb = wid * 512 + (lane & 31) * 16;
    int st = bb >> 10, sb = bb & 1023, sw = sb ^ (((sb >> 9) & 1) << 5);
    int R = (st >> 1) * 16 + (sw >> 6), C = (st & 1) * 32 + ((sw & 63) >> 1);
    xo = (unsigned)(R * (GK * 2) + C * 2);
  }
  auto stage_part = [&](int t, int s, int j) {
    if (j < MI)
      __builtin_amdgcn_global_load_lds((const unsigned*)(Abase + (size_t)t * (BK * 2) + aoff[j < MI ? j : 0]),
                                       (unsigned*)(smem + s * STB + (wid * MI + j) * 1024 + lane * 16), 16, 0, 0);
    else if (j < MI + 2)
      __builtin_amdgcn_global_load_lds((const unsigned*)(Bbase + (size_t)t * BK * (NL * 2) + boff[j == MI ? 0 : 1]),
                                       (unsigned*)(smem + s * STB + AB + (wid * 2 + j - MI) * 1024 + lane * 16), 16, 0, 0);
    else if (EX && lane < 32)
      __builtin_amdgcn_global_load_lds((const unsigned*)(Xbase + (size_t)t * (BK * 2) + xo), (unsigned*)(smem + s * STB + MI * 8192 + wid * 512 + lane * 16), 16, 0, 0);
  };
  auto stage = [&](int t, int s) {
#pragma unroll
    for (int j = 0; j < NLD; ++j) stage_part(t, s, j);
  };
  const unsigned lds0 = LDS_ADDR(smem);
  const unsigned cnt_addr = lds0 + CTRL_OFF;
  unsigned rdy = 0;
  auto wait_chunk = [&](int c) {
    if (rdy > (unsigned)c) return;
    if (wid != 0) {
      for (;;) {
        unsigned r;
        asm volatile("ds_read_b32 %0, %1 offset:4\n\ts_waitcnt lgkmcnt(0)" : "=v"(r) : "v"(cnt_addr) : "memory");
        rdy = __builtin_amdgcn_readfirstlane(r);
        if (rdy > (unsigned)c) break;
        __builtin_amdgcn_s_sleep(1);
      }
      return;
    }
    unsigned long long t0 = __builtin_amdgcn_s_memrealtime();
    unsigned nr;
    for (;;) {
      unsigned f = __hip_atomic_load(flagp + (lane & (NCH - 1)), __ATOMIC_RELAXED, __HIP_MEMORY_SCOPE_SYSTEM);
      unsigned long long m = ~__builtin_amdgcn_ballot_w64(f >= call);
      nr = m ? (unsigned)__builtin_ctzll(m) : 64u;
      if (nr > (unsigned)c) break;
      if (__hip_atomic_load(tmop, __ATOMIC_RELAXED, __HIP_MEMORY_SCOPE_AGENT) == call ||
          __builtin_amdgcn_s_memrealtime() - t0 > DEADLINE_TICKS) {
        __hip_atomic_store(tmop, call, __ATOMIC_RELAXED, __HIP_MEMORY_SCOPE_AGENT);
        nr = 64u;
        break;
      }
      __builtin_amdgcn_s_sleep(2);
    }
    if (lane == 0) asm volatile("ds_write_b32 %0, %1 offset:4" :: "v"(cnt_addr), "v"(nr) : "memory");
    rdy = nr;
    if (c == 0) TS(2);
    if (c <= NCH / 2) TS(5);
    TS(6);
  };

  const int wm = wid >> 1, wn = wid & 1, fr = lane & 15, fq = lane >> 4;
  const int q = fr >> 2, p = lane & 3;
  const unsigned abase = lds0 + (unsigned)ldsA_byte(wm * (16 * MI) + fr, fq * 8);
  const unsigned ebase = lds0 + (unsigned)ldsA_byte(64 * MI + (wm >> 1) * 16 + fr, fq * 8);
  f32x4 eacc[2] = {(f32x4){0.f, 0.f, 0.f, 0.f}, (f32x4){0.f, 0.f, 0.f, 0.f}};
  unsigned bbase[4][2];
#pragma unroll
  for (int ni = 0; ni < 4; ++ni)
#pragma unroll
    for (int tt = 0; tt < 2; ++tt) {
      int row = 8 * fq + 4 * tt + q, ch = 2 * (wn * 4 + ni) + (p >> 1);
      bbase[ni][tt] = lds0 + AB + (unsigned)(256 * row + 16 * (ch ^ swzB(row)) + 8 * (p & 1));
    }
  f32x4 acc[MI][4];
#pragma unroll
  for (int i = 0; i < MI; ++i)
#pragma unroll
    for (int j = 0; j < 4; ++j) acc[i][j] = (f32x4){0.f, 0.f, 0.f, 0.f};

  bf16x8 af[2][MI], ef[2]; s16x4 bh[2][4][2];
  constexpr int NRD = 8 + MI + EX;
  auto read_part = [&](int kk, unsigned so, int j) {
    if (j < 8) {
      const int ni = j >> 1, tt = j & 1;
      unsigned ba = bbase[ni][tt] + so;
      if (kk == 0) asm volatile("ds_read_b64_tr_b16 %0, %1" : "=v"(bh[kk][ni][tt]) : "v"(ba) : "memory");
      else asm volatile("ds_read_b64_tr_b16 %0, %1 offset:8192" : "=v"(bh[kk][ni][tt]) : "v"(ba) : "memory");
    } else if (j < 8 + MI) {
      const int mi = j < 8 + MI ? j - 8 : 0;
      asm volatile("ds_read_b128 %0, %1 offset:%2" : "=v"(af[kk][mi]) : "v"(abase + so), "i"(mi * 2048 + kk * 1024) : "memory");
    } else if (EX) asm volatile("ds_read_b128 %0, %1 offset:%2" : "=v"(ef[kk]) : "v"(ebase + so), "i"(kk * 1024) : "memory");
  };
  auto read_frags = [&](int kk, unsigned so) {
#pragma unroll
    for (int j = 0; j < NRD; ++j) read_part(kk, so, j);
  };
  auto pin = [&](int kk) {
#pragma unroll
    for (int mi = 0; mi < MI; ++mi) asm volatile("" : "+v"(af[kk][mi]));
#pragma unroll
    for (int ni = 0; ni < 4; ++ni) asm volatile("" : "+v"(bh[kk][ni][0]), "+v"(bh[kk][ni][1]));
    if (EX) asm volatile("" : "+v"(ef[kk]));
  };
  auto mma = [&](int kk, bool ld, int tl, int sl, int j0, int j1, bool rd = false, unsigned rso = 0) {
    constexpr int NM = MI * 4;
#pragma unroll
    for (int i = 0; i < NM; ++i) {
      const int mi = i >> 2, ni = i & 3;
      acc[mi][ni] = __builtin_amdgcn_mfma_f32_16x16x32_bf16(__builtin_shufflevector(bh[kk][ni][0], bh[kk][ni][1], 0, 1, 2, 3, 4, 5, 6, 7), af[kk][mi], acc[mi][ni], 0, 0, 0);
      if (ld) {
#pragma unroll
        for (int j = j0; j < j1; ++j)
          if (((j - j0) * NM) / (j1 - j0) == i) { __builtin_amdgcn_sched_barrier(0); stage_part(tl, sl, j); __builtin_amdgcn_sched_barrier(0); }
      }
    }
    if (EX) {
      if (wm & 1) {
        eacc[0] = __builtin_amdgcn_mfma_f32_16x16x32_bf16(__builtin_shufflevector(bh[kk][2][0], bh[kk][2][1], 0, 1, 2, 3, 4, 5, 6, 7), ef[kk], eacc[0], 0, 0, 0);
        eacc[1] = __builtin_amdgcn_mfma_f32_16x16x32_bf16(__builtin_shufflevector(bh[kk][3][0], bh[kk][3][1], 0, 1, 2, 3, 4, 5, 6, 7), ef[kk], eacc[1], 0, 0, 0);
      } else {
        eacc[0] = __builtin_amdgcn_mfma_f32_16x16x32_bf16(__builtin_shufflevector(bh[kk][0][0], bh[kk][0][1], 0, 1, 2, 3, 4, 5, 6, 7), ef[kk], eacc[0], 0, 0, 0);
        eacc[1] = __builtin_amdgcn_mfma_f32_16x16x32_bf16(__builtin_shufflevector(bh[kk][1][0], bh[kk][1][1], 0, 1, 2, 3, 4, 5, 6, 7), ef[kk], eacc[1], 0, 0, 0);
      }
    }
    __builtin_amdgcn_sched_barrier(0);
  };
  auto arrive = [&]() { if (lane == 0) asm volatile("ds_add_u32 %0, %1" :: "v"(cnt_addr), "v"(1u) : "memory"); };
  auto sync_stage = [&](int t) {
    const unsigned target = cnt0 + 8u * (unsigned)(t + 1);
    for (;;) {
      unsigned v;
      asm volatile("ds_read_b32 %0, %1\n\ts_waitcnt lgkmcnt(0)" : "=v"(v) : "v"(cnt_addr) : "memory");
      if (__builtin_amdgcn_readfirstlane(v) >= target) break;
    }
  };

#pragma unroll
  for (int i = 0; i < D; ++i) {
    if (gated && (i % IPC) == 0) wait_chunk(i / IPC);
    stage(i, i);
  }
  asm volatile("s_waitcnt vmcnt(%0)" :: "i"(NLD * (D - 1)) : "memory");
  arrive();
#define NL1 4
  constexpr int L1 = NL1 < NLD ? NL1 : 0, L0 = NLD - L1;
  int s = 0;
  for (int t = 0; t < NT - D; ++t) {
    sync_stage(t);
    const bool now = true;
    if (gated && ((t + D) % IPC) == 0) wait_chunk((t + D) / IPC);
    int s2 = s + D; if (s2 >= NS) s2 -= NS;
    read_frags(0, (unsigned)(s * STB));
    read_frags(1, (unsigned)(s * STB));
    asm volatile("s_waitcnt lgkmcnt(%0)" :: "i"(8 + MI + EX) : "memory");
    pin(0);
    mma(0, now, t + D, s2, 0, L0, true, (unsigned)(s * STB));
    asm volatile("s_waitcnt lgkmcnt(0)" ::: "memory");
    pin(1);
    if (now) asm volatile("s_waitcnt vmcnt(%0)" :: "i"(NLD * (D - 2) + L0) : "memory");
    else asm volatile("s_waitcnt vmcnt(%0)" :: "i"(NLD * (D - 2)) : "memory");
    arrive();
    mma(1, now && L1 > 0, t + D, s2, L0, NLD);
    if (!now) { wait_chunk((t + D) / IPC); stage(t + D, s2); }
    s = s + 1; if (s >= NS) s = 0;
  }
  for (int t = NT - D; t < NT; ++t) {
    sync_stage(t);
    read_frags(0, (unsigned)(s * STB));
    read_frags(1, (unsigned)(s * STB));
    asm volatile("s_waitcnt lgkmcnt(%0)" :: "i"(8 + MI + EX) : "memory");
    pin(0);
    mma(0, false, 0, 0, 0, 1);
    asm volatile("s_waitcnt lgkmcnt(0)" ::: "memory");
    pin(1);
    if (D > 2 && t + D == NT) asm volatile("s_waitcnt vmcnt(%0)" :: "i"(NLD * (D > 2 ? D - 2 : 0)) : "memory");
    else if (D > 3 && t + D == NT + 1) asm volatile("s_waitcnt vmcnt(%0)" :: "i"(NLD * (D > 3 ? D - 3 : 0)) : "memory");
    else if (D > 4 && t + D == NT + 2) asm volatile("s_waitcnt vmcnt(%0)" :: "i"(NLD * (D > 4 ? D - 4 : 0)) : "memory");
    else asm volatile("s_waitcnt vmcnt(0)" ::: "memory");
    arrive();
    mma(1, false, 0, 0, 0, 1);
    s = s + 1; if (s >= NS) s = 0;
  }
  if (wid == 0 && gated) TS(3);

  const bool bad = gated && (__hip_atomic_load(tmop, __ATOMIC_RELAXED, __HIP_MEMORY_SCOPE_AGENT) == call);
  auto store_pair = [&](char* p, const f32x4& u, const f32x4& v, bool nan) {
    unsigned a0 = pack_bf16(gelu_f(u[0]), gelu_f(u[1])), a1 = pack_bf16(gelu_f(u[2]), gelu_f(u[3]));
    unsigned b0 = pack_bf16(gelu_f(v[0]), gelu_f(v[1])), b1 = pack_bf16(gelu_f(v[2]), gelu_f(v[3]));
    auto r0 = __builtin_amdgcn_permlane16_swap(a0, b0, false, false);
    auto r1 = __builtin_amdgcn_permlane16_swap(a1, b1, false, false);
    u32x4 o = {r0[0], r1[0], r0[1], r1[1]};
    if (nan) o = (u32x4){0x7FC07FC0u, 0x7FC07FC0u, 0x7FC07FC0u, 0x7FC07FC0u};
    *(u32x4*)p = o;
  };
  const int lcol = (fq & 1) * 16 + (fq & 2) * 4;
  char* ob = obase + ((size_t)(wm * (16 * MI) + fr) * NL + wn * 64 + lcol) * 2;
#pragma unroll
  for (int mi = 0; mi < MI; ++mi)
#pragma unroll
    for (int h = 0; h < 2; ++h) store_pair(ob + ((size_t)mi * 16 * NL + h * 32) * 2, acc[mi][2 * h], acc[mi][2 * h + 1], bad);
  if (EX) store_pair(xobase + ((size_t)((wm >> 1) * 16 + fr) * NL + wn * 64 + (wm & 1) * 32 + lcol) * 2, eacc[0], eacc[1], false);
}

__global__ __launch_bounds__(512, 1) void ag_gemm_kernel(Args a) {
  extern __shared__ __attribute__((aligned(16))) char smem[];
  const int tid = threadIdx.x, lane = tid & 63;
  const int wid = __builtin_amdgcn_readfirstlane(tid >> 6);
  const int b = blockIdx.x, rank = a.rank;
  char* myws = a.ws[rank];
  const int xcd = b & 7, idx = b >> 3;
  const bool is_comm = idx < NCW / 8;
  constexpr unsigned RPI = 512 / KC, LPR = KC / 8;
  constexpr unsigned IPCH = MS / RPI, NITEMS = NCH * IPCH;
  const unsigned lo = (unsigned)(lane / LPR) * (GK * 2) + (unsigned)(lane % LPR) * 16;
  unsigned it = (unsigned)((idx * 8 + xcd) * 8 + wid);
  unsigned off = (it & (IPCH - 1)) * (RPI * GK * 2) + (it / IPCH) * (KC * 2) + lo;
  u32x4 v0;
  if (is_comm) asm volatile("global_load_dwordx4 %0, %1, off" : "=v"(v0) : "v"(a.x + off) : "memory");
  volatile unsigned* ctl = (volatile unsigned*)(smem + CTRL_OFF);
  if (tid == 0) {
    unsigned* cp = (unsigned*)(myws + WS_CALL + (size_t)b * 128);
    unsigned c = __hip_atomic_load(cp, __ATOMIC_RELAXED, __HIP_MEMORY_SCOPE_AGENT) + 1u;
    __hip_atomic_store(cp, c, __ATOMIC_RELAXED, __HIP_MEMORY_SCOPE_AGENT);
    ctl[0] = 0; ctl[1] = 0; ctl[2] = c;
    if (!is_comm) {
      __builtin_amdgcn_fence(__ATOMIC_ACQUIRE, "");
      asm volatile("s_waitcnt vmcnt(0)" ::: "memory");
    }
  }
  __syncthreads();
  const unsigned call = __builtin_amdgcn_readfirstlane(ctl[2]);
  const size_t xoff = (call & 1u) ? XBUF_BYTES : 0;
  if (wid == 0) TS(0);

  if (is_comm) {
    const unsigned base = (call - 1u) * NITEMS - NCW * 8;
    char* headp = myws + WS_HEAD;
    const size_t dbase = xoff + (size_t)rank * MS * (GK * 2);
    char* pw = a.ws[0];
#pragma unroll
    for (int p = 1; p < NR; ++p) pw = (lane == p) ? a.ws[p] : pw;
    pw += WS_FLAGS + (size_t)rank * 256;
    char* dst[NR - 1];
#pragma unroll
    for (int j = 0; j < NR - 1; ++j) dst[j] = a.ws[(rank + 1 + (j + b * 8 + wid) % 7) & 7] + dbase;
    unsigned g = 0, tk = 0, pc = 0xffffffffu;
    while (it < NITEMS) {
      if (lane == 0) asm volatile("global_atomic_add %0, %1, %2, off sc0" : "=v"(g) : "v"(headp), "v"(1u) : "memory");
      asm volatile("s_waitcnt vmcnt(1)" : "+v"(v0), "+v"(tk) :: "memory");
      asm volatile("global_store_dwordx4 %0, %1, off sc0 sc1" :: "v"(dst[0] + off), "v"(v0) : "memory");
      if (pc != 0xffffffffu && (__builtin_amdgcn_readfirstlane(tk) & (IPCH - 1)) == IPCH - 1 && lane < NR && lane != rank)
        asm volatile("global_store_dword %0, %1, off sc0 sc1" :: "v"(pw + (size_t)pc * 4), "v"(call) : "memory");
#pragma unroll
      for (int j = 1; j < NR - 1; ++j) {
        asm volatile("s_waitcnt vmcnt(%0)" :: "i"(INFL - 1) : "memory");
        asm volatile("global_store_dwordx4 %0, %1, off sc0 sc1" :: "v"(dst[j] + off), "v"(v0) : "memory");
      }
      asm volatile("s_waitcnt vmcnt(6)" : "+v"(g) :: "memory");
      pc = it / IPCH;
      it = __builtin_amdgcn_readfirstlane(g) - base;
      off = (it & (IPCH - 1)) * (RPI * GK * 2) + (it / IPCH) * (KC * 2) + lo;
      asm volatile("s_waitcnt vmcnt(0)" ::: "memory");
      if (lane == 0) asm volatile("global_atomic_add %0, %1, %2, off sc0" : "=v"(tk) : "v"(myws + WS_TICK + (size_t)pc * 128), "v"(1u) : "memory");
      if (it < NITEMS) asm volatile("global_load_dwordx4 %0, %1, off" : "=v"(v0) : "v"(a.x + off) : "memory");
    }
    asm volatile("s_waitcnt vmcnt(0)" : "+v"(tk) :: "memory");
    if (pc != 0xffffffffu && (__builtin_amdgcn_readfirstlane(tk) & (IPCH - 1)) == IPCH - 1 && lane < NR && lane != rank)
      asm volatile("global_store_dword %0, %1, off sc0 sc1" :: "v"(pw + (size_t)pc * 4), "v"(call) : "memory");
    if (wid == 0) TS(1);
    return;
  }

  unsigned* tmop = (unsigned*)(myws + WS_TMO);
#define TSARGS
  if (idx < NCW / 8 + 2) {
    const int nblk = xcd * 2 + (idx - NCW / 8);
    run_tile<1, 0>(smem, a.x + (size_t)448 * (GK * 2), a.w + (size_t)nblk * BN * 2, a.out + ((size_t)(rank * MS + 448) * NL + nblk * BN) * 2,
                   nullptr, nullptr, false, nullptr, tmop, call, 0u, wid, lane TSARGS);
    if (wid == 0) TS(7);
    return;
  }
  const int gi = idx - (NCW / 8 + 2);
  const int mi7 = gi % 7, nblk = (xcd & 3) * 4 + gi / 7;
  const int mg = (xcd >> 2) * 7 + mi7;
  const int mblk = (2 * (rank + 1) + mg) & 15;
  const int src = mblk >> 1;
  unsigned* flagp = (unsigned*)(myws + WS_FLAGS + (size_t)src * 256);
  {
    unsigned pf0, pf1 = 0;
    const char* wp = a.w + (size_t)nblk * BN * 2 + (size_t)(tid >> 1) * (NL * 2) + (tid & 1) * 128;
    asm volatile("global_load_dword %0, %1, off" : "=v"(pf0) : "v"(wp + (size_t)mg * 256 * (NL * 2)) : "memory");
    if (mg < 2) asm volatile("global_load_dword %0, %1, off" : "=v"(pf1) : "v"(wp + (size_t)(14 + mg) * 256 * (NL * 2)) : "memory");
    asm volatile("s_waitcnt vmcnt(0)" : "+v"(pf0), "+v"(pf1) :: "memory");
  }
  run_tile<4, 1>(smem, myws + xoff + (size_t)mblk * BM * (GK * 2), a.w + (size_t)nblk * BN * 2, a.out + ((size_t)mblk * BM * NL + nblk * BN) * 2,
                 a.x + (size_t)mg * 32 * (GK * 2), a.out + ((size_t)(rank * MS + mg * 32) * NL + nblk * BN) * 2,
                 true, flagp, tmop, call, 0u, wid, lane TSARGS);
}

extern "C" void kernel_launch(int rank, int nranks, ncclComm_t comm,
                              hipStream_t stream,
                              void* const* d_in, const int* in_sizes, int n_in,
                              void* d_out, int out_size,
                              const int* in_dtypes, int out_dtype,
                              void* const* d_out_all,
                              void* const* d_ws_all, size_t ws_bytes) {
  if (nranks != NR || ws_bytes < WS_NEED || (unsigned)rank >= (unsigned)NR) { fprintf(stderr, "unexpected configuration\n"); return; }
  static bool init = false;
  if (!init) {
    HIP_CHECK(hipFuncSetAttribute((const void*)ag_gemm_kernel, hipFuncAttributeMaxDynamicSharedMemorySize, LDS_BYTES));
    init = true;
  }
  Args a;
  a.x = (const char*)d_in[0]; a.w = (const char*)d_in[1]; a.out = (char*)d_out; a.rank = rank;
  for (int p = 0; p < NR; ++p) a.ws[p] = (char*)d_ws_all[p];
  ag_gemm_kernel<<<NWG, 512, LDS_BYTES, stream>>>(a);
  HIP_CHECK(hipGetLastError());
}
```

```cpp
#include <hip/hip_runtime.h>
#include <rccl/rccl.h>
#include <cstdio>
#include <cstdint>

#define HIP_CHECK(expr) do { hipError_t _e = (expr); if (_e != hipSuccess) { \
    fprintf(stderr, "HIP error %s at %s:%d\n", hipGetErrorString(_e), __FILE__, __LINE__); \
    return; } } while (0)

typedef __attribute__((ext_vector_type(8))) short bf16x8;
typedef __attribute__((ext_vector_type(4))) short s16x4;
typedef __attribute__((ext_vector_type(4))) float f32x4;
typedef __attribute__((ext_vector_type(4))) unsigned u32x4;
typedef __attribute__((ext_vector_type(2))) unsigned u32x2;

constexpr int NR = 8;
constexpr int GM = 4096, GK = 4096, NL = 2048, MS = 512;
constexpr int BM = 256, BN = 128, BK = 64, NT = GK / BK;
#define NCHUNK 32
constexpr int NCH = NCHUNK, KC = GK / NCH, IPC = KC / BK;
constexpr int NWG = 256, NCW = 16;
constexpr int NSTAGE = 3;
constexpr int A_BYTES = BM * BK * 2, B_BYTES = BK * BN * 2, ST_BYTES = A_BYTES + B_BYTES;
constexpr int CTRL_OFF = NSTAGE * (ST_BYTES + 4096);
constexpr int LDS_BYTES = CTRL_OFF + 64;

constexpr size_t XBUF_BYTES = (size_t)GM * GK * 2;
constexpr size_t WS_CTRL = 2 * XBUF_BYTES;
constexpr size_t WS_FLAGS = WS_CTRL;
constexpr size_t WS_TICK = WS_CTRL + 16384;
constexpr size_t WS_TMO = WS_CTRL + 24576;
constexpr size_t WS_HEAD = WS_CTRL + 24576 + 1024;
#define INFL 7
constexpr size_t WS_CALL = WS_CTRL + 32768;
constexpr size_t WS_DBG = WS_CALL + (size_t)NWG * 128;
constexpr size_t WS_NEED = WS_DBG + (size_t)NWG * 64;
#define TS(k) do {} while (0)
constexpr unsigned long long DEADLINE_TICKS = 50000000ull;

struct Args {
  const char* x; const char* w; char* out; char* ws[NR]; int rank;
};

__device__ __forceinline__ int ldsA_byte(int r, int c) {
  int st = (r >> 4) * 2 + (c >> 5), ob = (r & 15) * 64 + (c & 31) * 2;
  return st * 1024 + (ob ^ (((ob >> 9) & 1) << 5));
}
__device__ __forceinline__ int swzB(int row) { return ((row & 3) << 2) | ((row >> 2) & 3); }

__device__ __forceinline__ float gelu_f(float y) {
  const float c2 = 2.0f * 0.7978845608028654f;
  float u = c2 * (y + 0.044715f * y * y * y);
  return y * __builtin_amdgcn_rcpf(1.0f + __expf(-u));
}
__device__ __forceinline__ unsigned pack_bf16(float a, float b) {
  typedef __attribute__((ext_vector_type(2))) __bf16 b2;
  b2 v; v[0] = (__bf16)a; v[1] = (__bf16)b;
  return __builtin_bit_cast(unsigned, v);
}

#define LDS_ADDR(p) ((unsigned)(size_t)(__attribute__((address_space(3))) char*)(p))

template <int MI, int XR>
__device__ __forceinline__ void run_tile(char* smem, const char* Abase, const char* Bbase, char* obase, const char* Xbase, char* xobase, bool gated,
                                         unsigned* flagp, unsigned* tmop, unsigned call, unsigned cnt0, int wid, int lane
                                         ) {
#define MSP 2
  constexpr int MSPLIT = MI == 1 ? 0 : MSP;
  constexpr int EX = XR ? 1 : 0;
  const bool hasx = XR == 32 || (XR == 16 && wid < 4);
  constexpr int NLD = MI + 2 + EX;
  constexpr int AB = MI * 8192 + XR * 128, STB = AB + B_BYTES, NS = CTRL_OFF / STB, D = NS - 1;
  unsigned aoff[MI], boff[2];
#pragma unroll
  for (int i = 0; i < MI; ++i) {
    int bb = (wid * MI + i) * 1024 + lane * 16;
    int st = bb >> 10, sb = bb & 1023, sw = sb ^ (((sb >> 9) & 1) << 5);
    int R = (st >> 1) * 16 + (sw >> 6), C = (st & 1) * 32 + ((sw & 63) >> 1);
    aoff[i] = (unsigned)(R * (GK * 2) + C * 2);
  }
#pragma unroll
  for (int i = 0; i < 2; ++i) {
    int bb = (wid * 2 + i) * 1024 + lane * 16;
    int row = bb >> 8, pos = (bb >> 4) & 15, ch = pos ^ swzB(row);
    boff[i] = (unsigned)(row * (NL * 2) + ch * 16);
  }
  unsigned xo = 0;
  if (EX) {
    int bb = wid * (XR * 16) + (lane & (XR - 1)) * 16;
    int st = bb >> 10, sb = bb & 1023, sw = sb ^ (((sb >> 9) & 1) << 5);
    int R = (st >> 1) * 16 + (sw >> 6), C = (st & 1) * 32 + ((sw & 63) >> 1);
    xo = (unsigned)(R * (GK * 2) + C * 2);
  }
  auto stage_part = [&](int t, int s, int j) {
    if (j < MI)
      __builtin_amdgcn_global_load_lds((const unsigned*)(Abase + (size_t)t * (BK * 2) + aoff[j < MI ? j : 0]),
                                       (unsigned*)(smem + s * STB + (wid * MI + j) * 1024 + lane * 16), 16, 0, 0);
    else if (j < MI + 2)
      __builtin_amdgcn_global_load_lds((const unsigned*)(Bbase + (size_t)t * BK * (NL * 2) + boff[j == MI ? 0 : 1]),
                                       (unsigned*)(smem + s * STB + AB + (wid * 2 + j - MI) * 1024 + lane * 16), 16, 0, 0);
    else if (EX && lane < XR)
      __builtin_amdgcn_global_load_lds((const unsigned*)(Xbase + (size_t)t * (BK * 2) + xo), (unsigned*)(smem + s * STB + MI * 8192 + wid * (XR * 16) + lane * 16), 16, 0, 0);
  };
  auto stage = [&](int t, int s) {
#pragma unroll
    for (int j = 0; j < NLD; ++j) stage_part(t, s, j);
  };
  const unsigned lds0 = LDS_ADDR(smem);
  const unsigned cnt_addr = lds0 + CTRL_OFF;
  unsigned rdy = 0;
  auto wait_chunk = [&](int c) {
    if (rdy > (unsigned)c) return;
    if (wid != 0) {
      for (;;) {
        unsigned r;
        asm volatile("ds_read_b32 %0, %1 offset:4\n\ts_waitcnt lgkmcnt(0)" : "=v"(r) : "v"(cnt_addr) : "memory");
        rdy = __builtin_amdgcn_readfirstlane(r);
        if (rdy > (unsigned)c) break;
        __builtin_amdgcn_s_sleep(1);
      }
      return;
    }
    unsigned long long t0 = __builtin_amdgcn_s_memrealtime();
    unsigned nr;
    for (;;) {
      unsigned f = __hip_atomic_load(flagp + (lane & (NCH - 1)), __ATOMIC_RELAXED, __HIP_MEMORY_SCOPE_SYSTEM);
      unsigned long long m = ~__builtin_amdgcn_ballot_w64(f >= call);
      nr = m ? (unsigned)__builtin_ctzll(m) : 64u;
      if (nr > (unsigned)c) break;
      if (__hip_atomic_load(tmop, __ATOMIC_RELAXED, __HIP_MEMORY_SCOPE_AGENT) == call ||
          __builtin_amdgcn_s_memrealtime() - t0 > DEADLINE_TICKS) {
        __hip_atomic_store(tmop, call, __ATOMIC_RELAXED, __HIP_MEMORY_SCOPE_AGENT);
        nr = 64u;
        break;
      }
      __builtin_amdgcn_s_sleep(2);
    }
    if (lane == 0) asm volatile("ds_write_b32 %0, %1 offset:4" :: "v"(cnt_addr), "v"(nr) : "memory");
    rdy = nr;
    if (c == 0) TS(2);
    if (c <= NCH / 2) TS(5);
    TS(6);
  };

  const int wm = wid >> 1, wn = wid & 1, fr = lane & 15, fq = lane >> 4;
  const int q = fr >> 2, p = lane & 3;
  const unsigned abase = lds0 + (unsigned)ldsA_byte(wm * (16 * MI) + fr, fq * 8);
  const unsigned ebase = lds0 + (unsigned)ldsA_byte(64 * MI + (wm >> 1) * 16 + fr, fq * 8);
  f32x4 eacc[2] = {(f32x4){0.f, 0.f, 0.f, 0.f}, (f32x4){0.f, 0.f, 0.f, 0.f}};
  unsigned bbase[4][2];
#pragma unroll
  for (int ni = 0; ni < 4; ++ni)
#pragma unroll
    for (int tt = 0; tt < 2; ++tt) {
      int row = 8 * fq + 4 * tt + q, ch = 2 * (wn * 4 + ni) + (p >> 1);
      bbase[ni][tt] = lds0 + AB + (unsigned)(256 * row + 16 * (ch ^ swzB(row)) + 8 * (p & 1));
    }
  f32x4 acc[MI][4];
#pragma unroll
  for (int i = 0; i < MI; ++i)
#pragma unroll
    for (int j = 0; j < 4; ++j) acc[i][j] = (f32x4){0.f, 0.f, 0.f, 0.f};

  bf16x8 af[2][MI], ef[2]; s16x4 bh[2][4][2];
  constexpr int NRD = 8 + MI + EX;
  auto read_part = [&](int kk, unsigned so, int j) {
    if (j < 8) {
      const int ni = j >> 1, tt = j & 1;
      unsigned ba = bbase[ni][tt] + so;
      if (kk == 0) asm volatile("ds_read_b64_tr_b16 %0, %1" : "=v"(bh[kk][ni][tt]) : "v"(ba) : "memory");
      else asm volatile("ds_read_b64_tr_b16 %0, %1 offset:8192" : "=v"(bh[kk][ni][tt]) : "v"(ba) : "memory");
    } else if (j < 8 + MI) {
      const int mi = j < 8 + MI ? j - 8 : 0;
      asm volatile("ds_read_b128 %0, %1 offset:%2" : "=v"(af[kk][mi]) : "v"(abase + so), "i"(mi * 2048 + kk * 1024) : "memory");
    } else if (EX && hasx) asm volatile("ds_read_b128 %0, %1 offset:%2" : "=v"(ef[kk]) : "v"(ebase + so), "i"(kk * 1024) : "memory");
  };
  auto read_frags = [&](int kk, unsigned so) {
#pragma unroll
    for (int j = 0; j < NRD; ++j) read_part(kk, so, j);
  };
  auto pin = [&](int kk) {
#pragma unroll
    for (int mi = 0; mi < MI; ++mi) asm volatile("" : "+v"(af[kk][mi]));
#pragma unroll
    for (int ni = 0; ni < 4; ++ni) asm volatile("" : "+v"(bh[kk][ni][0]), "+v"(bh[kk][ni][1]));
    if (EX) asm volatile("" : "+v"(ef[kk]));
  };
  auto mma = [&](int kk, bool ld, int tl, int sl, int j0, int j1, bool rd = false, unsigned rso = 0) {
    constexpr int NM = MI * 4;
#pragma unroll
    for (int i = 0; i < NM; ++i) {
      const int mi = i >> 2, ni = i & 3;
      acc[mi][ni] = __builtin_amdgcn_mfma_f32_16x16x32_bf16(__builtin_shufflevector(bh[kk][ni][0], bh[kk][ni][1], 0, 1, 2, 3, 4, 5, 6, 7), af[kk][mi], acc[mi][ni], 0, 0, 0);
      if (ld) {
#pragma unroll
        for (int j = j0; j < j1; ++j)
          if (((j - j0) * NM) / (j1 - j0) == i) { __builtin_amdgcn_sched_barrier(0); stage_part(tl, sl, j); __builtin_amdgcn_sched_barrier(0); }
      }
    }
    if (EX && hasx) {
      if (wm & 1) {
        eacc[0] = __builtin_amdgcn_mfma_f32_16x16x32_bf16(__builtin_shufflevector(bh[kk][2][0], bh[kk][2][1], 0, 1, 2, 3, 4, 5, 6, 7), ef[kk], eacc[0], 0, 0, 0);
        eacc[1] = __builtin_amdgcn_mfma_f32_16x16x32_bf16(__builtin_shufflevector(bh[kk][3][0], bh[kk][3][1], 0, 1, 2, 3, 4, 5, 6, 7), ef[kk], eacc[1], 0, 0, 0);
      } else {
        eacc[0] = __builtin_amdgcn_mfma_f32_16x16x32_bf16(__builtin_shufflevector(bh[kk][0][0], bh[kk][0][1], 0, 1, 2, 3, 4, 5, 6, 7), ef[kk], eacc[0], 0, 0, 0);
        eacc[1] = __builtin_amdgcn_mfma_f32_16x16x32_bf16(__builtin_shufflevector(bh[kk][1][0], bh[kk][1][1], 0, 1, 2, 3, 4, 5, 6, 7), ef[kk], eacc[1], 0, 0, 0);
      }
    }
    __builtin_amdgcn_sched_barrier(0);
  };
  auto arrive = [&]() { if (lane == 0) asm volatile("ds_add_u32 %0, %1" :: "v"(cnt_addr), "v"(1u) : "memory"); };
  auto sync_stage = [&](int t) {
    const unsigned target = cnt0 + 8u * (unsigned)(t + 1);
    for (;;) {
      unsigned v;
      asm volatile("ds_read_b32 %0, %1\n\ts_waitcnt lgkmcnt(0)" : "=v"(v) : "v"(cnt_addr) : "memory");
      if (__builtin_amdgcn_readfirstlane(v) >= target) break;
    }
  };

#pragma unroll
  for (int i = 0; i < D; ++i) {
    if (gated && (i % IPC) == 0) wait_chunk(i / IPC);
    stage(i, i);
  }
  asm volatile("s_waitcnt vmcnt(%0)" :: "i"(NLD * (D - 1)) : "memory");
  arrive();
#define NL1 4
  constexpr int L1 = NL1 < NLD ? NL1 : 0, L0 = NLD - L1;
  int s = 0;
  for (int t = 0; t < NT - D; ++t) {
    sync_stage(t);
    const bool now = true;
    if (gated && ((t + D) % IPC) == 0) wait_chunk((t + D) / IPC);
    int s2 = s + D; if (s2 >= NS) s2 -= NS;
    read_frags(0, (unsigned)(s * STB));
    read_frags(1, (unsigned)(s * STB));
    asm volatile("s_waitcnt lgkmcnt(%0)" :: "i"(8 + MI + (XR == 32 ? 1 : 0)) : "memory");
    pin(0);
    mma(0, now, t + D, s2, 0, L0, true, (unsigned)(s * STB));
    asm volatile("s_waitcnt lgkmcnt(0)" ::: "memory");
    pin(1);
    if (now) asm volatile("s_waitcnt vmcnt(%0)" :: "i"(NLD * (D - 2) + L0) : "memory");
    else asm volatile("s_waitcnt vmcnt(%0)" :: "i"(NLD * (D - 2)) : "memory");
    arrive();
    mma(1, now && L1 > 0, t + D, s2, L0, NLD);
    if (!now) { wait_chunk((t + D) / IPC); stage(t + D, s2); }
    s = s + 1; if (s >= NS) s = 0;
  }
  for (int t = NT - D; t < NT; ++t) {
    sync_stage(t);
    read_frags(0, (unsigned)(s * STB));
    read_frags(1, (unsigned)(s * STB));
    asm volatile("s_waitcnt lgkmcnt(%0)" :: "i"(8 + MI + (XR == 32 ? 1 : 0)) : "memory");
    pin(0);
    mma(0, false, 0, 0, 0, 1);
    asm volatile("s_waitcnt lgkmcnt(0)" ::: "memory");
    pin(1);
    if (D > 2 && t + D == NT) asm volatile("s_waitcnt vmcnt(%0)" :: "i"(NLD * (D > 2 ? D - 2 : 0)) : "memory");
    else if (D > 3 && t + D == NT + 1) asm volatile("s_waitcnt vmcnt(%0)" :: "i"(NLD * (D > 3 ? D - 3 : 0)) : "memory");
    else if (D > 4 && t + D == NT + 2) asm volatile("s_waitcnt vmcnt(%0)" :: "i"(NLD * (D > 4 ? D - 4 : 0)) : "memory");
    else asm volatile("s_waitcnt vmcnt(0)" ::: "memory");
    arrive();
    mma(1, false, 0, 0, 0, 1);
    s = s + 1; if (s >= NS) s = 0;
  }
  if (wid == 0 && gated) TS(3);

  const bool bad = gated && (__hip_atomic_load(tmop, __ATOMIC_RELAXED, __HIP_MEMORY_SCOPE_AGENT) == call);
  auto store_pair = [&](char* p, const f32x4& u, const f32x4& v, bool nan) {
    unsigned a0 = pack_bf16(gelu_f(u[0]), gelu_f(u[1])), a1 = pack_bf16(gelu_f(u[2]), gelu_f(u[3]));
    unsigned b0 = pack_bf16(gelu_f(v[0]), gelu_f(v[1])), b1 = pack_bf16(gelu_f(v[2]), gelu_f(v[3]));
    auto r0 = __builtin_amdgcn_permlane16_swap(a0, b0, false, false);
    auto r1 = __builtin_amdgcn_permlane16_swap(a1, b1, false, false);
    u32x4 o = {r0[0], r1[0], r0[1], r1[1]};
    if (nan) o = (u32x4){0x7FC07FC0u, 0x7FC07FC0u, 0x7FC07FC0u, 0x7FC07FC0u};
    *(u32x4*)p = o;
  };
  const int lcol = (fq & 1) * 16 + (fq & 2) * 4;
  char* ob = obase + ((size_t)(wm * (16 * MI) + fr) * NL + wn * 64 + lcol) * 2;
#pragma unroll
  for (int mi = 0; mi < MI; ++mi)
#pragma unroll
    for (int h = 0; h < 2; ++h) store_pair(ob + ((size_t)mi * 16 * NL + h * 32) * 2, acc[mi][2 * h], acc[mi][2 * h + 1], bad);
  if (EX && hasx) store_pair(xobase + ((size_t)((wm >> 1) * 16 + fr) * NL + wn * 64 + (wm & 1) * 32 + lcol) * 2, eacc[0], eacc[1], false);
}

__global__ __launch_bounds__(512, 1) void ag_gemm_kernel(Args a) {
  extern __shared__ __attribute__((aligned(16))) char smem[];
  const int tid = threadIdx.x, lane = tid & 63;
  const int wid = __builtin_amdgcn_readfirstlane(tid >> 6);
  const int b = blockIdx.x, rank = a.rank;
  char* myws = a.ws[rank];
  const int xcd = b & 7, idx = b >> 3;
  const bool is_comm = idx < NCW / 8;
  constexpr unsigned RPI = 512 / KC, LPR = KC / 8;
  constexpr unsigned IPCH = MS / RPI, NITEMS = NCH * IPCH;
  const unsigned lo = (unsigned)(lane / LPR) * (GK * 2) + (unsigned)(lane % LPR) * 16;
  unsigned it = (unsigned)((idx * 8 + xcd) * 8 + wid);
  unsigned off = (it & (IPCH - 1)) * (RPI * GK * 2) + (it / IPCH) * (KC * 2) + lo;
  u32x4 v0;
  if (is_comm) asm volatile("global_load_dwordx4 %0, %1, off" : "=v"(v0) : "v"(a.x + off) : "memory");
  volatile unsigned* ctl = (volatile unsigned*)(smem + CTRL_OFF);
  if (tid == 0) {
    unsigned* cp = (unsigned*)(myws + WS_CALL + (size_t)b * 128);
    unsigned c = __hip_atomic_load(cp, __ATOMIC_RELAXED, __HIP_MEMORY_SCOPE_AGENT) + 1u;
    __hip_atomic_store(cp, c, __ATOMIC_RELAXED, __HIP_MEMORY_SCOPE_AGENT);
    ctl[0] = 0; ctl[1] = 0; ctl[2] = c;
    if (!is_comm) {
      __builtin_amdgcn_fence(__ATOMIC_ACQUIRE, "");
      asm volatile("s_waitcnt vmcnt(0)" ::: "memory");
    }
  }
  __syncthreads();
  const unsigned call = __builtin_amdgcn_readfirstlane(ctl[2]);
  const size_t xoff = (call & 1u) ? XBUF_BYTES : 0;
  if (wid == 0) TS(0);

  if (is_comm) {
    const unsigned base = (call - 1u) * NITEMS - NCW * 8;
    char* headp = myws + WS_HEAD;
    const size_t dbase = xoff + (size_t)rank * MS * (GK * 2);
    char* pw = a.ws[0];
#pragma unroll
    for (int p = 1; p < NR; ++p) pw = (lane == p) ? a.ws[p] : pw;
    pw += WS_FLAGS + (size_t)rank * 256;
    char* dst[NR - 1];
#pragma unroll
    for (int j = 0; j < NR - 1; ++j) dst[j] = a.ws[(rank + 1 + (j + b * 8 + wid) % 7) & 7] + dbase;
    unsigned g = 0, tk = 0, pc = 0xffffffffu;
    while (it < NITEMS) {
      if (lane == 0) asm volatile("global_atomic_add %0, %1, %2, off sc0" : "=v"(g) : "v"(headp), "v"(1u) : "memory");
      asm volatile("s_waitcnt vmcnt(1)" : "+v"(v0), "+v"(tk) :: "memory");
      asm volatile("global_store_dwordx4 %0, %1, off sc0 sc1" :: "v"(dst[0] + off), "v"(v0) : "memory");
      if (pc != 0xffffffffu && (__builtin_amdgcn_readfirstlane(tk) & (IPCH - 1)) == IPCH - 1 && lane < NR && lane != rank)
        asm volatile("global_store_dword %0, %1, off sc0 sc1" :: "v"(pw + (size_t)pc * 4), "v"(call) : "memory");
#pragma unroll
      for (int j = 1; j < NR - 1; ++j) {
        asm volatile("s_waitcnt vmcnt(%0)" :: "i"(INFL - 1) : "memory");
        asm volatile("global_store_dwordx4 %0, %1, off sc0 sc1" :: "v"(dst[j] + off), "v"(v0) : "memory");
      }
      asm volatile("s_waitcnt vmcnt(6)" : "+v"(g) :: "memory");
      pc = it / IPCH;
      it = __builtin_amdgcn_readfirstlane(g) - base;
      off = (it & (IPCH - 1)) * (RPI * GK * 2) + (it / IPCH) * (KC * 2) + lo;
      asm volatile("s_waitcnt vmcnt(0)" ::: "memory");
      if (lane == 0) asm volatile("global_atomic_add %0, %1, %2, off sc0" : "=v"(tk) : "v"(myws + WS_TICK + (size_t)pc * 128), "v"(1u) : "memory");
      if (it < NITEMS) asm volatile("global_load_dwordx4 %0, %1, off" : "=v"(v0) : "v"(a.x + off) : "memory");
    }
    asm volatile("s_waitcnt vmcnt(0)" : "+v"(tk) :: "memory");
    if (pc != 0xffffffffu && (__builtin_amdgcn_readfirstlane(tk) & (IPCH - 1)) == IPCH - 1 && lane < NR && lane != rank)
      asm volatile("global_store_dword %0, %1, off sc0 sc1" :: "v"(pw + (size_t)pc * 4), "v"(call) : "memory");
    if (wid == 0) TS(1);
    return;
  }

  unsigned* tmop = (unsigned*)(myws + WS_TMO);
#define TSARGS
  if (idx < NCW / 8 + 2) {
    const int nblk = xcd * 2 + (idx - NCW / 8);
    run_tile<4, 32>(smem, a.x + (size_t)224 * (GK * 2), a.w + (size_t)nblk * BN * 2, a.out + ((size_t)(rank * MS + 224) * NL + nblk * BN) * 2,
                    a.x + (size_t)480 * (GK * 2), a.out + ((size_t)(rank * MS + 480) * NL + nblk * BN) * 2, false, nullptr, tmop, call, 0u, wid, lane TSARGS);
    if (wid == 0) TS(7);
    return;
  }
  const int gi = idx - (NCW / 8 + 2);
  const int mi7 = gi % 7, nblk = (xcd & 3) * 4 + gi / 7;
  const int mg = (xcd >> 2) * 7 + mi7;
  const int mblk = (2 * (rank + 1) + mg) & 15;
  const int src = mblk >> 1;
  unsigned* flagp = (unsigned*)(myws + WS_FLAGS + (size_t)src * 256);
  run_tile<4, 16>(smem, myws + xoff + (size_t)mblk * BM * (GK * 2), a.w + (size_t)nblk * BN * 2, a.out + ((size_t)mblk * BM * NL + nblk * BN) * 2,
                 a.x + (size_t)mg * 16 * (GK * 2), a.out + ((size_t)(rank * MS + mg * 16) * NL + nblk * BN) * 2,
                 true, flagp, tmop, call, 0u, wid, lane TSARGS);
}

extern "C" void kernel_launch(int rank, int nranks, ncclComm_t comm,
                              hipStream_t stream,
                              void* const* d_in, const int* in_sizes, int n_in,
                              void* d_out, int out_size,
                              const int* in_dtypes, int out_dtype,
                              void* const* d_out_all,
                              void* const* d_ws_all, size_t ws_bytes) {
  if (nranks != NR || ws_bytes < WS_NEED || (unsigned)rank >= (unsigned)NR) { fprintf(stderr, "unexpected configuration\n"); return; }
  static bool init = false;
  if (!init) {
    HIP_CHECK(hipFuncSetAttribute((const void*)ag_gemm_kernel, hipFuncAttributeMaxDynamicSharedMemorySize, LDS_BYTES));
    init = true;
  }
  Args a;
  a.x = (const char*)d_in[0]; a.w = (const char*)d_in[1]; a.out = (char*)d_out; a.rank = rank;
  for (int p = 0; p < NR; ++p) a.ws[p] = (char*)d_ws_all[p];
  ag_gemm_kernel<<<NWG, 512, LDS_BYTES, stream>>>(a);
  HIP_CHECK(hipGetLastError());
}
```

```cpp
#include <hip/hip_runtime.h>
#include <rccl/rccl.h>
#include <cstdio>
#include <cstdint>

#define HIP_CHECK(expr) do { hipError_t _e = (expr); if (_e != hipSuccess) { \
    fprintf(stderr, "HIP error %s at %s:%d\n", hipGetErrorString(_e), __FILE__, __LINE__); \
    return; } } while (0)

typedef __attribute__((ext_vector_type(8))) short bf16x8;
typedef __attribute__((ext_vector_type(4))) short s16x4;
typedef __attribute__((ext_vector_type(4))) float f32x4;
typedef __attribute__((ext_vector_type(4))) unsigned u32x4;
typedef __attribute__((ext_vector_type(2))) unsigned u32x2;

#define COMM2 3
#define PROBE 1
constexpr int NR = 8;
constexpr int GM = 4096, GK = 4096, NL = 2048, MS = 512;
constexpr int BM = 256, BN = 128, BK = 64, NT = GK / BK;
#define NCHUNK 32
constexpr int NCH = NCHUNK, KC = GK / NCH, IPC = KC / BK;
constexpr int NWG = 256, NCW = 16;
constexpr int NSTAGE = 3;
constexpr int A_BYTES = BM * BK * 2, B_BYTES = BK * BN * 2, ST_BYTES = A_BYTES + B_BYTES;
constexpr int CTRL_OFF = NSTAGE * (ST_BYTES + 4096);
constexpr int LDS_BYTES = CTRL_OFF + 64;

constexpr size_t XBUF_BYTES = (size_t)GM * GK * 2;
constexpr size_t WS_CTRL = 2 * XBUF_BYTES;
constexpr size_t WS_FLAGS = WS_CTRL;
constexpr size_t WS_TICK = WS_CTRL + 16384;
constexpr size_t WS_TMO = WS_CTRL + 24576;
constexpr size_t WS_HEAD = WS_CTRL + 24576 + 1024;
#define INFL 7
constexpr size_t WS_CALL = WS_CTRL + 32768;
constexpr size_t WS_DBG = WS_CALL + (size_t)NWG * 128;
constexpr size_t WS_NEED = WS_DBG + (size_t)NWG * 64;
#define TS(k) do {} while (0)
constexpr unsigned long long DEADLINE_TICKS = 50000000ull;

struct Args {
  const char* x; const char* w; char* out; char* ws[NR]; int rank;
};

__device__ __forceinline__ int ldsA_byte(int r, int c) {
  int st = (r >> 4) * 2 + (c >> 5), ob = (r & 15) * 64 + (c & 31) * 2;
  return st * 1024 + (ob ^ (((ob >> 9) & 1) << 5));
}
__device__ __forceinline__ int swzB(int row) { return ((row & 3) << 2) | ((row >> 2) & 3); }

__device__ __forceinline__ float gelu_f(float y) {
  const float c2 = 2.0f * 0.7978845608028654f;
  float u = c2 * (y + 0.044715f * y * y * y);
  return y * __builtin_amdgcn_rcpf(1.0f + __expf(-u));
}
__device__ __forceinline__ unsigned pack_bf16(float a, float b) {
  typedef __attribute__((ext_vector_type(2))) __bf16 b2;
  b2 v; v[0] = (__bf16)a; v[1] = (__bf16)b;
  return __builtin_bit_cast(unsigned, v);
}

#define LDS_ADDR(p) ((unsigned)(size_t)(__attribute__((address_space(3))) char*)(p))

template <int MI, int XR>
__device__ __forceinline__ void run_tile(char* smem, const char* Abase, const char* Bbase, char* obase, const char* Xbase, char* xobase, bool gated,
                                         unsigned* flagp, unsigned* tmop, unsigned call, unsigned cnt0, int wid, int lane
                                         ) {
#define MSP 2
  constexpr int MSPLIT = MI == 1 ? 0 : MSP;
  constexpr int EX = XR ? 1 : 0;
  const bool hasx = XR == 32 || (XR == 16 && wid < 4);
  constexpr int NLD = MI + 2 + EX;
  constexpr int AB = MI * 8192 + XR * 128, STB = AB + B_BYTES, NS = CTRL_OFF / STB, D = NS - 1;
  unsigned aoff[MI], boff[2];
#pragma unroll
  for (int i = 0; i < MI; ++i) {
    int bb = (wid * MI + i) * 1024 + lane * 16;
    int st = bb >> 10, sb = bb & 1023, sw = sb ^ (((sb >> 9) & 1) << 5);
    int R = (st >> 1) * 16 + (sw >> 6), C = (st & 1) * 32 + ((sw & 63) >> 1);
    aoff[i] = (unsigned)(R * (GK * 2) + C * 2);
  }
#pragma unroll
  for (int i = 0; i < 2; ++i) {
    int bb = (wid * 2 + i) * 1024 + lane * 16;
    int row = bb >> 8, pos = (bb >> 4) & 15, ch = pos ^ swzB(row);
    boff[i] = (unsigned)(row * (NL * 2) + ch * 16);
  }
  unsigned xo = 0;
  if (EX) {
    int bb = wid * (XR * 16) + (lane & (XR - 1)) * 16;
    int st = bb >> 10, sb = bb & 1023, sw = sb ^ (((sb >> 9) & 1) << 5);
    int R = (st >> 1) * 16 + (sw >> 6), C = (st & 1) * 32 + ((sw & 63) >> 1);
    xo = (unsigned)(R * (GK * 2) + C * 2);
  }
  auto stage_part = [&](int t, int s, int j) {
    if (j < MI)
      __builtin_amdgcn_global_load_lds((const unsigned*)(Abase + (size_t)t * (BK * 2) + aoff[j < MI ? j : 0]),
                                       (unsigned*)(smem + s * STB + (wid * MI + j) * 1024 + lane * 16), 16, 0, 0);
    else if (j < MI + 2)
      __builtin_amdgcn_global_load_lds((const unsigned*)(Bbase + (size_t)t * BK * (NL * 2) + boff[j == MI ? 0 : 1]),
                                       (unsigned*)(smem + s * STB + AB + (wid * 2 + j - MI) * 1024 + lane * 16), 16, 0, 0);
    else if (EX && lane < XR)
      __builtin_amdgcn_global_load_lds((const unsigned*)(Xbase + (size_t)t * (BK * 2) + xo), (unsigned*)(smem + s * STB + MI * 8192 + wid * (XR * 16) + lane * 16), 16, 0, 0);
  };
  auto stage = [&](int t, int s) {
#pragma unroll
    for (int j = 0; j < NLD; ++j) stage_part(t, s, j);
  };
  const unsigned lds0 = LDS_ADDR(smem);
  const unsigned cnt_addr = lds0 + CTRL_OFF;
  unsigned rdy = 0;
  auto wait_chunk = [&](int c) {
    if (rdy > (unsigned)c) return;
    if (wid != 0) {
      for (;;) {
        unsigned r;
        asm volatile("ds_read_b32 %0, %1 offset:4\n\ts_waitcnt lgkmcnt(0)" : "=v"(r) : "v"(cnt_addr) : "memory");
        rdy = __builtin_amdgcn_readfirstlane(r);
        if (rdy > (unsigned)c) break;
        __builtin_amdgcn_s_sleep(1);
      }
      return;
    }
    unsigned long long t0 = __builtin_amdgcn_s_memrealtime();
    unsigned nr;
    for (;;) {
      unsigned f = __hip_atomic_load(flagp + (lane & (NCH - 1)), __ATOMIC_RELAXED, __HIP_MEMORY_SCOPE_SYSTEM);
      unsigned long long m = ~__builtin_amdgcn_ballot_w64(f >= call);
      nr = m ? (unsigned)__builtin_ctzll(m) : 64u;
      if (nr > (unsigned)c) break;
      if (__hip_atomic_load(tmop, __ATOMIC_RELAXED, __HIP_MEMORY_SCOPE_AGENT) == call ||
          __builtin_amdgcn_s_memrealtime() - t0 > DEADLINE_TICKS) {
        __hip_atomic_store(tmop, call, __ATOMIC_RELAXED, __HIP_MEMORY_SCOPE_AGENT);
        nr = 64u;
        break;
      }
      __builtin_amdgcn_s_sleep(2);
    }
    if (lane == 0) asm volatile("ds_write_b32 %0, %1 offset:4" :: "v"(cnt_addr), "v"(nr) : "memory");
    rdy = nr;
    if (c == 0) TS(2);
    if (c <= NCH / 2) TS(5);
    TS(6);
  };

  const int wm = wid >> 1, wn = wid & 1, fr = lane & 15, fq = lane >> 4;
  const int q = fr >> 2, p = lane & 3;
  const unsigned abase = lds0 + (unsigned)ldsA_byte(wm * (16 * MI) + fr, fq * 8);
  const unsigned ebase = lds0 + (unsigned)ldsA_byte(64 * MI + (wm >> 1) * 16 + fr, fq * 8);
  f32x4 eacc[2] = {(f32x4){0.f, 0.f, 0.f, 0.f}, (f32x4){0.f, 0.f, 0.f, 0.f}};
  unsigned bbase[4][2];
#pragma unroll
  for (int ni = 0; ni < 4; ++ni)
#pragma unroll
    for (int tt = 0; tt < 2; ++tt) {
      int row = 8 * fq + 4 * tt + q, ch = 2 * (wn * 4 + ni) + (p >> 1);
      bbase[ni][tt] = lds0 + AB + (unsigned)(256 * row + 16 * (ch ^ swzB(row)) + 8 * (p & 1));
    }
  f32x4 acc[MI][4];
#pragma unroll
  for (int i = 0; i < MI; ++i)
#pragma unroll
    for (int j = 0; j < 4; ++j) acc[i][j] = (f32x4){0.f, 0.f, 0.f, 0.f};

  bf16x8 af[2][MI], ef[2]; s16x4 bh[2][4][2];
  constexpr int NRD = 8 + MI + EX;
  auto read_part = [&](int kk, unsigned so, int j) {
    if (j < 8) {
      const int ni = j >> 1, tt = j & 1;
      unsigned ba = bbase[ni][tt] + so;
      if (kk == 0) asm volatile("ds_read_b64_tr_b16 %0, %1" : "=v"(bh[kk][ni][tt]) : "v"(ba) : "memory");
      else asm volatile("ds_read_b64_tr_b16 %0, %1 offset:8192" : "=v"(bh[kk][ni][tt]) : "v"(ba) : "memory");
    } else if (j < 8 + MI) {
      const int mi = j < 8 + MI ? j - 8 : 0;
      asm volatile("ds_read_b128 %0, %1 offset:%2" : "=v"(af[kk][mi]) : "v"(abase + so), "i"(mi * 2048 + kk * 1024) : "memory");
    } else if (EX && hasx) asm volatile("ds_read_b128 %0, %1 offset:%2" : "=v"(ef[kk]) : "v"(ebase + so), "i"(kk * 1024) : "memory");
  };
  auto read_frags = [&](int kk, unsigned so) {
#pragma unroll
    for (int j = 0; j < NRD; ++j) read_part(kk, so, j);
  };
  auto pin = [&](int kk) {
#pragma unroll
    for (int mi = 0; mi < MI; ++mi) asm volatile("" : "+v"(af[kk][mi]));
#pragma unroll
    for (int ni = 0; ni < 4; ++ni) asm volatile("" : "+v"(bh[kk][ni][0]), "+v"(bh[kk][ni][1]));
    if (EX) asm volatile("" : "+v"(ef[kk]));
  };
  auto mma = [&](int kk, bool ld, int tl, int sl, int j0, int j1, bool rd = false, unsigned rso = 0) {
    constexpr int NM = MI * 4;
#pragma unroll
    for (int i = 0; i < NM; ++i) {
      const int mi = i >> 2, ni = i & 3;
      acc[mi][ni] = __builtin_amdgcn_mfma_f32_16x16x32_bf16(__builtin_shufflevector(bh[kk][ni][0], bh[kk][ni][1], 0, 1, 2, 3, 4, 5, 6, 7), af[kk][mi], acc[mi][ni], 0, 0, 0);
      if (ld) {
#pragma unroll
        for (int j = j0; j < j1; ++j)
          if (((j - j0) * NM) / (j1 - j0) == i) { __builtin_amdgcn_sched_barrier(0); stage_part(tl, sl, j); __builtin_amdgcn_sched_barrier(0); }
      }
    }
    if (EX && hasx) {
      if (wm & 1) {
        eacc[0] = __builtin_amdgcn_mfma_f32_16x16x32_bf16(__builtin_shufflevector(bh[kk][2][0], bh[kk][2][1], 0, 1, 2, 3, 4, 5, 6, 7), ef[kk], eacc[0], 0, 0, 0);
        eacc[1] = __builtin_amdgcn_mfma_f32_16x16x32_bf16(__builtin_shufflevector(bh[kk][3][0], bh[kk][3][1], 0, 1, 2, 3, 4, 5, 6, 7), ef[kk], eacc[1], 0, 0, 0);
      } else {
        eacc[0] = __builtin_amdgcn_mfma_f32_16x16x32_bf16(__builtin_shufflevector(bh[kk][0][0], bh[kk][0][1], 0, 1, 2, 3, 4, 5, 6, 7), ef[kk], eacc[0], 0, 0, 0);
        eacc[1] = __builtin_amdgcn_mfma_f32_16x16x32_bf16(__builtin_shufflevector(bh[kk][1][0], bh[kk][1][1], 0, 1, 2, 3, 4, 5, 6, 7), ef[kk], eacc[1], 0, 0, 0);
      }
    }
    __builtin_amdgcn_sched_barrier(0);
  };
  auto arrive = [&]() { if (lane == 0) asm volatile("ds_add_u32 %0, %1" :: "v"(cnt_addr), "v"(1u) : "memory"); };
  auto sync_stage = [&](int t) {
    const unsigned target = cnt0 + 8u * (unsigned)(t + 1);
    for (;;) {
      u32x2 v;
      asm volatile("ds_read_b64 %0, %1\n\ts_waitcnt lgkmcnt(0)" : "=v"(v) : "v"(cnt_addr) : "memory");
      if (__builtin_amdgcn_readfirstlane(v[0]) >= target) {
        unsigned r = __builtin_amdgcn_readfirstlane(v[1]);
        rdy = r > rdy ? r : rdy;
        break;
      }
    }
  };

#pragma unroll
  for (int i = 0; i < D; ++i) {
    if (gated && (i % IPC) == 0) wait_chunk(i / IPC);
    stage(i, i);
  }
  asm volatile("s_waitcnt vmcnt(%0)" :: "i"(NLD * (D - 1)) : "memory");
  arrive();
#define NL1 4
  constexpr int L1 = NL1 < NLD ? NL1 : 0, L0 = NLD - L1;
  int s = 0;
  for (int t = 0; t < NT - D; ++t) {
    sync_stage(t);
    unsigned f = 0;
    const bool probe = gated && wid == 0 && rdy < (unsigned)NCH;
    if (probe) asm volatile("global_load_dword %0, %1, off sc0 sc1" : "=v"(f) : "v"(flagp + (lane & (NCH - 1))) : "memory");
    if (gated && ((t + D) % IPC) == 0) wait_chunk((t + D) / IPC);
    int s2 = s + D; if (s2 >= NS) s2 -= NS;
    read_frags(0, (unsigned)(s * STB));
    read_frags(1, (unsigned)(s * STB));
    asm volatile("s_waitcnt lgkmcnt(%0)" :: "i"(8 + MI + (XR == 32 ? 1 : 0)) : "memory");
    pin(0);
    mma(0, true, t + D, s2, 0, L0, true, (unsigned)(s * STB));
    asm volatile("s_waitcnt lgkmcnt(0)" ::: "memory");
    pin(1);
    asm volatile("s_waitcnt vmcnt(%0)" :: "i"(NLD * (D - 2) + L0) : "memory");
    asm volatile("" : "+v"(f));
    if (probe) {
      unsigned long long m = ~__builtin_amdgcn_ballot_w64(f >= call);
      unsigned nr = m ? (unsigned)__builtin_ctzll(m) : 64u;
      if (nr > rdy) {
        rdy = nr;
        if (lane == 0) asm volatile("ds_write_b32 %0, %1 offset:4" :: "v"(cnt_addr), "v"(nr) : "memory");
      }
    }
    arrive();
    mma(1, L1 > 0, t + D, s2, L0, NLD);
    s = s + 1; if (s >= NS) s = 0;
  }
  for (int t = NT - D; t < NT; ++t) {
    sync_stage(t);
    read_frags(0, (unsigned)(s * STB));
    read_frags(1, (unsigned)(s * STB));
    asm volatile("s_waitcnt lgkmcnt(%0)" :: "i"(8 + MI + (XR == 32 ? 1 : 0)) : "memory");
    pin(0);
    mma(0, false, 0, 0, 0, 1);
    asm volatile("s_waitcnt lgkmcnt(0)" ::: "memory");
    pin(1);
    if (D > 2 && t + D == NT) asm volatile("s_waitcnt vmcnt(%0)" :: "i"(NLD * (D > 2 ? D - 2 : 0)) : "memory");
    else if (D > 3 && t + D == NT + 1) asm volatile("s_waitcnt vmcnt(%0)" :: "i"(NLD * (D > 3 ? D - 3 : 0)) : "memory");
    else if (D > 4 && t + D == NT + 2) asm volatile("s_waitcnt vmcnt(%0)" :: "i"(NLD * (D > 4 ? D - 4 : 0)) : "memory");
    else asm volatile("s_waitcnt vmcnt(0)" ::: "memory");
    arrive();
    mma(1, false, 0, 0, 0, 1);
    s = s + 1; if (s >= NS) s = 0;
  }
  if (wid == 0 && gated) TS(3);

  const bool bad = gated && (__hip_atomic_load(tmop, __ATOMIC_RELAXED, __HIP_MEMORY_SCOPE_AGENT) == call);
  auto store_pair = [&](char* p, const f32x4& u, const f32x4& v, bool nan) {
    unsigned a0 = pack_bf16(gelu_f(u[0]), gelu_f(u[1])), a1 = pack_bf16(gelu_f(u[2]), gelu_f(u[3]));
    unsigned b0 = pack_bf16(gelu_f(v[0]), gelu_f(v[1])), b1 = pack_bf16(gelu_f(v[2]), gelu_f(v[3]));
    auto r0 = __builtin_amdgcn_permlane16_swap(a0, b0, false, false);
    auto r1 = __builtin_amdgcn_permlane16_swap(a1, b1, false, false);
    u32x4 o = {r0[0], r1[0], r0[1], r1[1]};
    if (nan) o = (u32x4){0x7FC07FC0u, 0x7FC07FC0u, 0x7FC07FC0u, 0x7FC07FC0u};
    *(u32x4*)p = o;
  };
  const int lcol = (fq & 1) * 16 + (fq & 2) * 4;
  char* ob = obase + ((size_t)(wm * (16 * MI) + fr) * NL + wn * 64 + lcol) * 2;
#pragma unroll
  for (int mi = 0; mi < MI; ++mi)
#pragma unroll
    for (int h = 0; h < 2; ++h) store_pair(ob + ((size_t)mi * 16 * NL + h * 32) * 2, acc[mi][2 * h], acc[mi][2 * h + 1], bad);
  if (EX && hasx) store_pair(xobase + ((size_t)((wm >> 1) * 16 + fr) * NL + wn * 64 + (wm & 1) * 32 + lcol) * 2, eacc[0], eacc[1], false);
}

__global__ __launch_bounds__(512, 1) void ag_gemm_kernel(Args a) {
  extern __shared__ __attribute__((aligned(16))) char smem[];
  const int tid = threadIdx.x, lane = tid & 63;
  const int wid = __builtin_amdgcn_readfirstlane(tid >> 6);
  const int b = blockIdx.x, rank = a.rank;
  char* myws = a.ws[rank];
  const int xcd = b & 7, idx = b >> 3;
  const bool is_comm = idx < NCW / 8;
  constexpr unsigned RPI = 512 / KC, LPR = KC / 8;
  constexpr unsigned IPCH = MS / RPI, NITEMS = NCH * IPCH;
  const unsigned lo = (unsigned)(lane / LPR) * (GK * 2) + (unsigned)(lane % LPR) * 16;
  unsigned it = (unsigned)((idx * 8 + xcd) * 8 + wid);
  unsigned off = (it & (IPCH - 1)) * (RPI * GK * 2) + (it / IPCH) * (KC * 2) + lo;
  u32x4 v0;
  if (is_comm) asm volatile("global_load_dwordx4 %0, %1, off" : "=v"(v0) : "v"(a.x + off) : "memory");
  volatile unsigned* ctl = (volatile unsigned*)(smem + CTRL_OFF);
  if (tid == 0) {
    unsigned* cp = (unsigned*)(myws + WS_CALL + (size_t)b * 128);
    unsigned c = __hip_atomic_load(cp, __ATOMIC_RELAXED, __HIP_MEMORY_SCOPE_AGENT) + 1u;
    __hip_atomic_store(cp, c, __ATOMIC_RELAXED, __HIP_MEMORY_SCOPE_AGENT);
    ctl[0] = 0; ctl[1] = 0; ctl[2] = c;
    if (!is_comm) {
      __builtin_amdgcn_fence(__ATOMIC_ACQUIRE, "");
      asm volatile("s_waitcnt vmcnt(0)" ::: "memory");
    }
  }
  __syncthreads();
  const unsigned call = __builtin_amdgcn_readfirstlane(ctl[2]);
  const size_t xoff = (call & 1u) ? XBUF_BYTES : 0;
  if (wid == 0) TS(0);

  if (is_comm) {
    const unsigned base = (call - 1u) * NITEMS - NCW * 8;
    char* headp = myws + WS_HEAD;
    const size_t dbase = xoff + (size_t)rank * MS * (GK * 2);
    char* pw = a.ws[0];
#pragma unroll
    for (int p = 1; p < NR; ++p) pw = (lane == p) ? a.ws[p] : pw;
    pw += WS_FLAGS + (size_t)rank * 256;
    char* dst[NR - 1];
#pragma unroll
    for (int j = 0; j < NR - 1; ++j) dst[j] = a.ws[(rank + 1 + (j + b * 8 + wid) % 7) & 7] + dbase;
    unsigned g = 0, tk = 0, pc = 0xffffffffu;
    asm volatile("s_waitcnt vmcnt(0)" : "+v"(v0) :: "memory");
    while (it < NITEMS) {
      if (lane == 0) asm volatile("global_atomic_add %0, %1, %2, off sc0" : "=v"(g) : "v"(headp), "v"(1u) : "memory");
#pragma unroll
      for (int j = 0; j < COMM2; ++j)
        asm volatile("global_store_dwordx4 %0, %1, off sc0 sc1" :: "v"(dst[j] + off), "v"(v0) : "memory");
      asm volatile("s_waitcnt vmcnt(%2)" : "+v"(g), "+v"(tk) : "i"(COMM2) : "memory");
      if (pc != 0xffffffffu && (__builtin_amdgcn_readfirstlane(tk) & (IPCH - 1)) == IPCH - 1 && lane < NR && lane != rank)
        asm volatile("global_store_dword %0, %1, off sc0 sc1" :: "v"(pw + (size_t)pc * 4), "v"(call) : "memory");
#pragma unroll
      for (int j = COMM2; j < NR - 1; ++j)
        asm volatile("global_store_dwordx4 %0, %1, off sc0 sc1" :: "v"(dst[j] + off), "v"(v0) : "memory");
      pc = it / IPCH;
      it = __builtin_amdgcn_readfirstlane(g) - base;
      off = (it & (IPCH - 1)) * (RPI * GK * 2) + (it / IPCH) * (KC * 2) + lo;
      if (it < NITEMS) asm volatile("global_load_dwordx4 %0, %1, off" : "=v"(v0) : "v"(a.x + off) : "memory");
      asm volatile("s_waitcnt vmcnt(0)" : "+v"(v0) :: "memory");
      if (lane == 0) asm volatile("global_atomic_add %0, %1, %2, off sc0" : "=v"(tk) : "v"(myws + WS_TICK + (size_t)pc * 128), "v"(1u) : "memory");
    }
    asm volatile("s_waitcnt vmcnt(0)" : "+v"(tk) :: "memory");
    if (pc != 0xffffffffu && (__builtin_amdgcn_readfirstlane(tk) & (IPCH - 1)) == IPCH - 1 && lane < NR && lane != rank)
      asm volatile("global_store_dword %0, %1, off sc0 sc1" :: "v"(pw + (size_t)pc * 4), "v"(call) : "memory");
    if (wid == 0) TS(1);
    return;
  }

  unsigned* tmop = (unsigned*)(myws + WS_TMO);
#define TSARGS
  if (idx < NCW / 8 + 2) {
    const int nblk = xcd * 2 + (idx - NCW / 8);
    run_tile<4, 32>(smem, a.x + (size_t)224 * (GK * 2), a.w + (size_t)nblk * BN * 2, a.out + ((size_t)(rank * MS + 224) * NL + nblk * BN) * 2,
                    a.x + (size_t)480 * (GK * 2), a.out + ((size_t)(rank * MS + 480) * NL + nblk * BN) * 2, false, nullptr, tmop, call, 0u, wid, lane TSARGS);
    if (wid == 0) TS(7);
    return;
  }
  const int gi = idx - (NCW / 8 + 2);
  const int mi7 = gi % 7, nblk = (xcd & 3) * 4 + gi / 7;
  const int mg = (xcd >> 2) * 7 + mi7;
  const int mblk = (2 * (rank + 1) + mg) & 15;
  const int src = mblk >> 1;
  unsigned* flagp = (unsigned*)(myws + WS_FLAGS + (size_t)src * 256);
  run_tile<4, 16>(smem, myws + xoff + (size_t)mblk * BM * (GK * 2), a.w + (size_t)nblk * BN * 2, a.out + ((size_t)mblk * BM * NL + nblk * BN) * 2,
                 a.x + (size_t)mg * 16 * (GK * 2), a.out + ((size_t)(rank * MS + mg * 16) * NL + nblk * BN) * 2,
                 true, flagp, tmop, call, 0u, wid, lane TSARGS);
}

extern "C" void kernel_launch(int rank, int nranks, ncclComm_t comm,
                              hipStream_t stream,
                              void* const* d_in, const int* in_sizes, int n_in,
                              void* d_out, int out_size,
                              const int* in_dtypes, int out_dtype,
                              void* const* d_out_all,
                              void* const* d_ws_all, size_t ws_bytes) {
  if (nranks != NR || ws_bytes < WS_NEED || (unsigned)rank >= (unsigned)NR) { fprintf(stderr, "unexpected configuration\n"); return; }
  static bool init = false;
  if (!init) {
    HIP_CHECK(hipFuncSetAttribute((const void*)ag_gemm_kernel, hipFuncAttributeMaxDynamicSharedMemorySize, LDS_BYTES));
    init = true;
  }
  Args a;
  a.x = (const char*)d_in[0]; a.w = (const char*)d_in[1]; a.out = (char*)d_out; a.rank = rank;
  for (int p = 0; p < NR; ++p) a.ws[p] = (char*)d_ws_all[p];
  ag_gemm_kernel<<<NWG, 512, LDS_BYTES, stream>>>(a);
  HIP_CHECK(hipGetLastError());
}
```

```cpp
#include <hip/hip_runtime.h>
#include <rccl/rccl.h>
#include <cstdio>
#include <cstdint>

#define HIP_CHECK(expr) do { hipError_t _e = (expr); if (_e != hipSuccess) { \
    fprintf(stderr, "HIP error %s at %s:%d\n", hipGetErrorString(_e), __FILE__, __LINE__); \
    return; } } while (0)

typedef __attribute__((ext_vector_type(8))) short bf16x8;
typedef __attribute__((ext_vector_type(4))) short s16x4;
typedef __attribute__((ext_vector_type(4))) float f32x4;
typedef __attribute__((ext_vector_type(4))) unsigned u32x4;
typedef __attribute__((ext_vector_type(2))) unsigned u32x2;

constexpr int NR = 8;
constexpr int GM = 4096, GK = 4096, NL = 2048, MS = 512;
constexpr int BM = 256, BN = 128, BK = 64, NT = GK / BK;
#define NCHUNK 32
constexpr int NCH = NCHUNK, KC = GK / NCH, IPC = KC / BK;
constexpr int NWG = 256, NCW = 16;
constexpr int NSTAGE = 3;
constexpr int FJ = 3;
constexpr int A_BYTES = BM * BK * 2, B_BYTES = BK * BN * 2, ST_BYTES = A_BYTES + B_BYTES;
constexpr int CTRL_OFF = NSTAGE * (ST_BYTES + 4096);
constexpr int LDS_BYTES = CTRL_OFF + 64;

constexpr size_t XBUF_BYTES = (size_t)GM * GK * 2;
constexpr size_t WS_CTRL = 2 * XBUF_BYTES;
constexpr size_t WS_FLAGS = WS_CTRL;
constexpr size_t WS_TICK = WS_CTRL + 16384;
constexpr size_t WS_TMO = WS_CTRL + 24576;
constexpr size_t WS_HEAD = WS_CTRL + 24576 + 1024;
constexpr size_t WS_CALL = WS_CTRL + 32768;
constexpr size_t WS_DBG = WS_CALL + (size_t)NWG * 128;
constexpr size_t WS_NEED = WS_DBG + (size_t)NWG * 64;
#define TS(k) do {} while (0)
constexpr unsigned long long DEADLINE_TICKS = 50000000ull;

struct Args {
  const char* x; const char* w; char* out; char* ws[NR]; int rank;
};

__device__ __forceinline__ int ldsA_byte(int r, int c) {
  int st = (r >> 4) * 2 + (c >> 5), ob = (r & 15) * 64 + (c & 31) * 2;
  return st * 1024 + (ob ^ (((ob >> 9) & 1) << 5));
}
__device__ __forceinline__ int swzB(int row) { return ((row & 3) << 2) | ((row >> 2) & 3); }

__device__ __forceinline__ float gelu_f(float y) {
  const float c2 = 2.0f * 0.7978845608028654f;
  float u = c2 * (y + 0.044715f * y * y * y);
  return y * __builtin_amdgcn_rcpf(1.0f + __expf(-u));
}
__device__ __forceinline__ unsigned pack_bf16(float a, float b) {
  typedef __attribute__((ext_vector_type(2))) __bf16 b2;
  b2 v; v[0] = (__bf16)a; v[1] = (__bf16)b;
  return __builtin_bit_cast(unsigned, v);
}

#define LDS_ADDR(p) ((unsigned)(size_t)(__attribute__((address_space(3))) char*)(p))

template <int MI, int XR>
__device__ __forceinline__ void run_tile(char* smem, const char* Abase, const char* Bbase, char* obase, const char* Xbase, char* xobase, bool gated,
                                         unsigned* flagp, unsigned* tmop, unsigned call, unsigned cnt0, int wid, int lane
                                         ) {
  constexpr int EX = XR ? 1 : 0;
  const bool hasx = XR == 32 || (XR == 16 && wid < 4);
  constexpr int NLD = MI + 2 + EX;
  constexpr int AB = MI * 8192 + XR * 128, STB = AB + B_BYTES, NS = CTRL_OFF / STB, D = NS - 1;
  unsigned aoff[MI], boff[2];
#pragma unroll
  for (int i = 0; i < MI; ++i) {
    int bb = (wid * MI + i) * 1024 + lane * 16;
    int st = bb >> 10, sb = bb & 1023, sw = sb ^ (((sb >> 9) & 1) << 5);
    int R = (st >> 1) * 16 + (sw >> 6), C = (st & 1) * 32 + ((sw & 63) >> 1);
    aoff[i] = (unsigned)(R * (GK * 2) + C * 2);
  }
#pragma unroll
  for (int i = 0; i < 2; ++i) {
    int bb = (wid * 2 + i) * 1024 + lane * 16;
    int row = bb >> 8, pos = (bb >> 4) & 15, ch = pos ^ swzB(row);
    boff[i] = (unsigned)(row * (NL * 2) + ch * 16);
  }
  unsigned xo = 0;
  if (EX) {
    int bb = wid * (XR * 16) + (lane & (XR - 1)) * 16;
    int st = bb >> 10, sb = bb & 1023, sw = sb ^ (((sb >> 9) & 1) << 5);
    int R = (st >> 1) * 16 + (sw >> 6), C = (st & 1) * 32 + ((sw & 63) >> 1);
    xo = (unsigned)(R * (GK * 2) + C * 2);
  }
  auto stage_part = [&](int t, int s, int j) {
    if (j < MI)
      __builtin_amdgcn_global_load_lds((const unsigned*)(Abase + (size_t)t * (BK * 2) + aoff[j < MI ? j : 0]),
                                       (unsigned*)(smem + s * STB + (wid * MI + j) * 1024 + lane * 16), 16, 0, 0);
    else if (j < MI + 2)
      __builtin_amdgcn_global_load_lds((const unsigned*)(Bbase + (size_t)t * BK * (NL * 2) + boff[j == MI ? 0 : 1]),
                                       (unsigned*)(smem + s * STB + AB + (wid * 2 + j - MI) * 1024 + lane * 16), 16, 0, 0);
    else if (EX && lane < XR)
      __builtin_amdgcn_global_load_lds((const unsigned*)(Xbase + (size_t)t * (BK * 2) + xo), (unsigned*)(smem + s * STB + MI * 8192 + wid * (XR * 16) + lane * 16), 16, 0, 0);
  };
  auto stage = [&](int t, int s) {
#pragma unroll
    for (int j = 0; j < NLD; ++j) stage_part(t, s, j);
  };
  const unsigned lds0 = LDS_ADDR(smem);
  const unsigned cnt_addr = lds0 + CTRL_OFF;
  unsigned rdy = 0;
  auto wait_chunk = [&](int c) {
    if (rdy > (unsigned)c) return;
    if (wid != 0) {
      for (;;) {
        unsigned r;
        asm volatile("ds_read_b32 %0, %1 offset:4\n\ts_waitcnt lgkmcnt(0)" : "=v"(r) : "v"(cnt_addr) : "memory");
        rdy = __builtin_amdgcn_readfirstlane(r);
        if (rdy > (unsigned)c) break;
        __builtin_amdgcn_s_sleep(1);
      }
      return;
    }
    unsigned long long t0 = __builtin_amdgcn_s_memrealtime();
    unsigned nr;
    for (;;) {
      unsigned f = __hip_atomic_load(flagp + (lane & (NCH - 1)), __ATOMIC_RELAXED, __HIP_MEMORY_SCOPE_SYSTEM);
      unsigned long long m = ~__builtin_amdgcn_ballot_w64(f >= call);
      nr = m ? (unsigned)__builtin_ctzll(m) : 64u;
      if (nr > (unsigned)c) break;
      if (__hip_atomic_load(tmop, __ATOMIC_RELAXED, __HIP_MEMORY_SCOPE_AGENT) == call ||
          __builtin_amdgcn_s_memrealtime() - t0 > DEADLINE_TICKS) {
        __hip_atomic_store(tmop, call, __ATOMIC_RELAXED, __HIP_MEMORY_SCOPE_AGENT);
        nr = 64u;
        break;
      }
      __builtin_amdgcn_s_sleep(2);
    }
    if (lane == 0) asm volatile("ds_write_b32 %0, %1 offset:4" :: "v"(cnt_addr), "v"(nr) : "memory");
    rdy = nr;
    if (c == 0) TS(2);
    if (c <= NCH / 2) TS(5);
    TS(6);
  };

  const int wm = wid >> 1, wn = wid & 1, fr = lane & 15, fq = lane >> 4;
  const int q = fr >> 2, p = lane & 3;
  const unsigned abase = lds0 + (unsigned)ldsA_byte(wm * (16 * MI) + fr, fq * 8);
  const unsigned ebase = lds0 + (unsigned)ldsA_byte(64 * MI + (wm >> 1) * 16 + fr, fq * 8);
  f32x4 eacc[2] = {(f32x4){0.f, 0.f, 0.f, 0.f}, (f32x4){0.f, 0.f, 0.f, 0.f}};
  unsigned bbase[4][2];
#pragma unroll
  for (int ni = 0; ni < 4; ++ni)
#pragma unroll
    for (int tt = 0; tt < 2; ++tt) {
      int row = 8 * fq + 4 * tt + q, ch = 2 * (wn * 4 + ni) + (p >> 1);
      bbase[ni][tt] = lds0 + AB + (unsigned)(256 * row + 16 * (ch ^ swzB(row)) + 8 * (p & 1));
    }
  f32x4 acc[MI][4];
#pragma unroll
  for (int i = 0; i < MI; ++i)
#pragma unroll
    for (int j = 0; j < 4; ++j) acc[i][j] = (f32x4){0.f, 0.f, 0.f, 0.f};

  bf16x8 af[2][MI], ef[2]; s16x4 bh[2][4][2];
  constexpr int NRD = 8 + MI + EX;
  auto read_part = [&](int kk, unsigned so, int j) {
    if (j < 8) {
      const int ni = j >> 1, tt = j & 1;
      unsigned ba = bbase[ni][tt] + so;
      if (kk == 0) asm volatile("ds_read_b64_tr_b16 %0, %1" : "=v"(bh[kk][ni][tt]) : "v"(ba) : "memory");
      else asm volatile("ds_read_b64_tr_b16 %0, %1 offset:8192" : "=v"(bh[kk][ni][tt]) : "v"(ba) : "memory");
    } else if (j < 8 + MI) {
      const int mi = j < 8 + MI ? j - 8 : 0;
      asm volatile("ds_read_b128 %0, %1 offset:%2" : "=v"(af[kk][mi]) : "v"(abase + so), "i"(mi * 2048 + kk * 1024) : "memory");
    } else if (EX && hasx) asm volatile("ds_read_b128 %0, %1 offset:%2" : "=v"(ef[kk]) : "v"(ebase + so), "i"(kk * 1024) : "memory");
  };
  auto read_frags = [&](int kk, unsigned so) {
#pragma unroll
    for (int j = 0; j < NRD; ++j) read_part(kk, so, j);
  };
  auto pin = [&](int kk) {
#pragma unroll
    for (int mi = 0; mi < MI; ++mi) asm volatile("" : "+v"(af[kk][mi]));
#pragma unroll
    for (int ni = 0; ni < 4; ++ni) asm volatile("" : "+v"(bh[kk][ni][0]), "+v"(bh[kk][ni][1]));
    if (EX) asm volatile("" : "+v"(ef[kk]));
  };
  auto mma = [&](int kk, bool ld, int tl, int sl, int j0, int j1) {
    constexpr int NM = MI * 4;
#pragma unroll
    for (int i = 0; i < NM; ++i) {
      const int mi = i >> 2, ni = i & 3;
      acc[mi][ni] = __builtin_amdgcn_mfma_f32_16x16x32_bf16(__builtin_shufflevector(bh[kk][ni][0], bh[kk][ni][1], 0, 1, 2, 3, 4, 5, 6, 7), af[kk][mi], acc[mi][ni], 0, 0, 0);
      if (ld) {
#pragma unroll
        for (int j = j0; j < j1; ++j)
          if (((j - j0) * NM) / (j1 - j0) == i) { __builtin_amdgcn_sched_barrier(0); stage_part(tl, sl, j); __builtin_amdgcn_sched_barrier(0); }
      }
    }
    if (EX && hasx) {
      if (wm & 1) {
        eacc[0] = __builtin_amdgcn_mfma_f32_16x16x32_bf16(__builtin_shufflevector(bh[kk][2][0], bh[kk][2][1], 0, 1, 2, 3, 4, 5, 6, 7), ef[kk], eacc[0], 0, 0, 0);
        eacc[1] = __builtin_amdgcn_mfma_f32_16x16x32_bf16(__builtin_shufflevector(bh[kk][3][0], bh[kk][3][1], 0, 1, 2, 3, 4, 5, 6, 7), ef[kk], eacc[1], 0, 0, 0);
      } else {
        eacc[0] = __builtin_amdgcn_mfma_f32_16x16x32_bf16(__builtin_shufflevector(bh[kk][0][0], bh[kk][0][1], 0, 1, 2, 3, 4, 5, 6, 7), ef[kk], eacc[0], 0, 0, 0);
        eacc[1] = __builtin_amdgcn_mfma_f32_16x16x32_bf16(__builtin_shufflevector(bh[kk][1][0], bh[kk][1][1], 0, 1, 2, 3, 4, 5, 6, 7), ef[kk], eacc[1], 0, 0, 0);
      }
    }
    __builtin_amdgcn_sched_barrier(0);
  };
  auto arrive = [&]() { if (lane == 0) asm volatile("ds_add_u32 %0, %1" :: "v"(cnt_addr), "v"(1u) : "memory"); };
  auto sync_stage = [&](int t) {
    const unsigned target = cnt0 + 8u * (unsigned)(t + 1);
    for (;;) {
      u32x2 v;
      asm volatile("ds_read_b64 %0, %1\n\ts_waitcnt lgkmcnt(0)" : "=v"(v) : "v"(cnt_addr) : "memory");
      if (__builtin_amdgcn_readfirstlane(v[0]) >= target) {
        unsigned r = __builtin_amdgcn_readfirstlane(v[1]);
        rdy = r > rdy ? r : rdy;
        break;
      }
    }
  };

  unsigned pt;
  asm volatile("global_load_dword %0, %1, off" : "=v"(pt) : "v"((wid & 1 ? xobase : obase + (size_t)wid * 32 * (NL * 2))) : "memory");
#pragma unroll
  for (int i = 0; i < D; ++i) {
    if (gated && (i % IPC) == 0) wait_chunk(i / IPC);
    stage(i, i);
  }
  asm volatile("s_waitcnt vmcnt(%1)" : "+v"(pt) : "i"(NLD * (D - 1)) : "memory");
  arrive();
#define NL1 4
  constexpr int L1 = NL1 < NLD ? NL1 : 0, L0 = NLD - L1;
  int s = 0;
  for (int t = 0; t < NT - D; ++t) {
    sync_stage(t);
    unsigned f = 0;
    const bool probe = gated && wid == 0 && rdy < (unsigned)NCH;
    if (probe) asm volatile("global_load_dword %0, %1, off sc0 sc1" : "=v"(f) : "v"(flagp + (lane & (NCH - 1))) : "memory");
    if (gated && ((t + D) % IPC) == 0) wait_chunk((t + D) / IPC);
    int s2 = s + D; if (s2 >= NS) s2 -= NS;
    read_frags(0, (unsigned)(s * STB));
    read_frags(1, (unsigned)(s * STB));
    asm volatile("s_waitcnt lgkmcnt(%0)" :: "i"(8 + MI + (XR == 32 ? 1 : 0)) : "memory");
    pin(0);
    mma(0, true, t + D, s2, 0, L0);
    asm volatile("s_waitcnt lgkmcnt(0)" ::: "memory");
    pin(1);
    asm volatile("s_waitcnt vmcnt(%0)" :: "i"(NLD * (D - 2) + L0) : "memory");
    asm volatile("" : "+v"(f));
    if (probe) {
      unsigned long long m = ~__builtin_amdgcn_ballot_w64(f >= call);
      unsigned nr = m ? (unsigned)__builtin_ctzll(m) : 64u;
      if (nr > rdy) {
        rdy = nr;
        if (lane == 0) asm volatile("ds_write_b32 %0, %1 offset:4" :: "v"(cnt_addr), "v"(nr) : "memory");
      }
    }
    arrive();
    mma(1, L1 > 0, t + D, s2, L0, NLD);
    s = s + 1; if (s >= NS) s = 0;
  }
  for (int t = NT - D; t < NT; ++t) {
    sync_stage(t);
    read_frags(0, (unsigned)(s * STB));
    read_frags(1, (unsigned)(s * STB));
    asm volatile("s_waitcnt lgkmcnt(%0)" :: "i"(8 + MI + (XR == 32 ? 1 : 0)) : "memory");
    pin(0);
    mma(0, false, 0, 0, 0, 1);
    asm volatile("s_waitcnt lgkmcnt(0)" ::: "memory");
    pin(1);
    if (D > 2 && t + D == NT) asm volatile("s_waitcnt vmcnt(%0)" :: "i"(NLD * (D > 2 ? D - 2 : 0)) : "memory");
    else if (D > 3 && t + D == NT + 1) asm volatile("s_waitcnt vmcnt(%0)" :: "i"(NLD * (D > 3 ? D - 3 : 0)) : "memory");
    else if (D > 4 && t + D == NT + 2) asm volatile("s_waitcnt vmcnt(%0)" :: "i"(NLD * (D > 4 ? D - 4 : 0)) : "memory");
    else asm volatile("s_waitcnt vmcnt(0)" ::: "memory");
    arrive();
    mma(1, false, 0, 0, 0, 1);
    s = s + 1; if (s >= NS) s = 0;
  }
  if (wid == 0 && gated) TS(3);

  const bool bad = gated && (__hip_atomic_load(tmop, __ATOMIC_RELAXED, __HIP_MEMORY_SCOPE_AGENT) == call);
  auto store_pair = [&](char* p, const f32x4& u, const f32x4& v, bool nan) {
    unsigned a0 = pack_bf16(gelu_f(u[0]), gelu_f(u[1])), a1 = pack_bf16(gelu_f(u[2]), gelu_f(u[3]));
    unsigned b0 = pack_bf16(gelu_f(v[0]), gelu_f(v[1])), b1 = pack_bf16(gelu_f(v[2]), gelu_f(v[3]));
    auto r0 = __builtin_amdgcn_permlane16_swap(a0, b0, false, false);
    auto r1 = __builtin_amdgcn_permlane16_swap(a1, b1, false, false);
    u32x4 o = {r0[0], r1[0], r0[1], r1[1]};
    if (nan) o = (u32x4){0x7FC07FC0u, 0x7FC07FC0u, 0x7FC07FC0u, 0x7FC07FC0u};
    __builtin_nontemporal_store(o, (u32x4*)p);
  };
  const int lcol = (fq & 1) * 16 + (fq & 2) * 4;
  char* ob = obase + ((size_t)(wm * (16 * MI) + fr) * NL + wn * 64 + lcol) * 2;
#pragma unroll
  for (int mi = 0; mi < MI; ++mi)
#pragma unroll
    for (int h = 0; h < 2; ++h) store_pair(ob + ((size_t)mi * 16 * NL + h * 32) * 2, acc[mi][2 * h], acc[mi][2 * h + 1], bad);
  if (EX && hasx) store_pair(xobase + ((size_t)((wm >> 1) * 16 + fr) * NL + wn * 64 + (wm & 1) * 32 + lcol) * 2, eacc[0], eacc[1], false);
}

__global__ __launch_bounds__(512, 1) void ag_gemm_kernel(Args a) {
  extern __shared__ __attribute__((aligned(16))) char smem[];
  const int tid = threadIdx.x, lane = tid & 63;
  const int wid = __builtin_amdgcn_readfirstlane(tid >> 6);
  const int b = blockIdx.x, rank = a.rank;
  char* myws = a.ws[rank];
  const int xcd = b & 7, idx = b >> 3;
  const bool is_comm = idx < NCW / 8;
  constexpr unsigned RPI = 512 / KC, LPR = KC / 8;
  constexpr unsigned IPCH = MS / RPI, NITEMS = NCH * IPCH;
  const unsigned lo = (unsigned)(lane / LPR) * (GK * 2) + (unsigned)(lane % LPR) * 16;
  unsigned it = (unsigned)((idx * 8 + xcd) * 8 + wid);
  unsigned off = (it & (IPCH - 1)) * (RPI * GK * 2) + (it / IPCH) * (KC * 2) + lo;
  u32x4 v0;
  if (is_comm) asm volatile("global_load_dwordx4 %0, %1, off" : "=v"(v0) : "v"(a.x + off) : "memory");
  volatile unsigned* ctl = (volatile unsigned*)(smem + CTRL_OFF);
  if (tid == 0) {
    unsigned* cp = (unsigned*)(myws + WS_CALL + (size_t)b * 128);
    unsigned c = __hip_atomic_load(cp, __ATOMIC_RELAXED, __HIP_MEMORY_SCOPE_AGENT) + 1u;
    __hip_atomic_store(cp, c, __ATOMIC_RELAXED, __HIP_MEMORY_SCOPE_AGENT);
    ctl[0] = 0; ctl[1] = 0; ctl[2] = c;
    if (!is_comm) {
      __builtin_amdgcn_fence(__ATOMIC_ACQUIRE, "");
      asm volatile("s_waitcnt vmcnt(0)" ::: "memory");
    }
  }
  __syncthreads();
  const unsigned call = __builtin_amdgcn_readfirstlane(ctl[2]);
  const size_t xoff = (call & 1u) ? XBUF_BYTES : 0;
  if (wid == 0) TS(0);

  if (is_comm) {
    const unsigned base = (call - 1u) * NITEMS - NCW * 8;
    char* headp = myws + WS_HEAD;
    const size_t dbase = xoff + (size_t)rank * MS * (GK * 2);
    char* pw = a.ws[0];
#pragma unroll
    for (int p = 1; p < NR; ++p) pw = (lane == p) ? a.ws[p] : pw;
    pw += WS_FLAGS + (size_t)rank * 256;
    char* dst[NR - 1];
#pragma unroll
    for (int j = 0; j < NR - 1; ++j) dst[j] = a.ws[(rank + 1 + (j + b * 8 + wid) % 7) & 7] + dbase;
    unsigned g = 0, tk = 0, pc = 0xffffffffu;
    asm volatile("s_waitcnt vmcnt(0)" : "+v"(v0) :: "memory");
    while (it < NITEMS) {
      if (lane == 0) asm volatile("global_atomic_add %0, %1, %2, off sc0" : "=v"(g) : "v"(headp), "v"(1u) : "memory");
#pragma unroll
      for (int j = 0; j < FJ; ++j)
        asm volatile("global_store_dwordx4 %0, %1, off sc0 sc1" :: "v"(dst[j] + off), "v"(v0) : "memory");
      asm volatile("s_waitcnt vmcnt(%2)" : "+v"(g), "+v"(tk) : "i"(FJ) : "memory");
      if (pc != 0xffffffffu && (__builtin_amdgcn_readfirstlane(tk) & (IPCH - 1)) == IPCH - 1 && lane < NR && lane != rank)
        asm volatile("global_store_dword %0, %1, off sc0 sc1" :: "v"(pw + (size_t)pc * 4), "v"(call) : "memory");
#pragma unroll
      for (int j = FJ; j < NR - 1; ++j)
        asm volatile("global_store_dwordx4 %0, %1, off sc0 sc1" :: "v"(dst[j] + off), "v"(v0) : "memory");
      pc = it / IPCH;
      it = __builtin_amdgcn_readfirstlane(g) - base;
      off = (it & (IPCH - 1)) * (RPI * GK * 2) + (it / IPCH) * (KC * 2) + lo;
      if (it < NITEMS) asm volatile("global_load_dwordx4 %0, %1, off" : "=v"(v0) : "v"(a.x + off) : "memory");
      asm volatile("s_waitcnt vmcnt(0)" : "+v"(v0) :: "memory");
      if (lane == 0) asm volatile("global_atomic_add %0, %1, %2, off sc0" : "=v"(tk) : "v"(myws + WS_TICK + (size_t)pc * 128), "v"(1u) : "memory");
    }
    asm volatile("s_waitcnt vmcnt(0)" : "+v"(tk) :: "memory");
    if (pc != 0xffffffffu && (__builtin_amdgcn_readfirstlane(tk) & (IPCH - 1)) == IPCH - 1 && lane < NR && lane != rank)
      asm volatile("global_store_dword %0, %1, off sc0 sc1" :: "v"(pw + (size_t)pc * 4), "v"(call) : "memory");
    if (wid == 0) TS(1);
    return;
  }

  unsigned* tmop = (unsigned*)(myws + WS_TMO);
#define TSARGS
  if (idx < NCW / 8 + 2) {
    const int nblk = xcd * 2 + (idx - NCW / 8);
    run_tile<4, 32>(smem, a.x + (size_t)224 * (GK * 2), a.w + (size_t)nblk * BN * 2, a.out + ((size_t)(rank * MS + 224) * NL + nblk * BN) * 2,
                    a.x + (size_t)480 * (GK * 2), a.out + ((size_t)(rank * MS + 480) * NL + nblk * BN) * 2, false, nullptr, tmop, call, 0u, wid, lane TSARGS);
    if (wid == 0) TS(7);
    return;
  }
  const int gi = idx - (NCW / 8 + 2);
  const int mi7 = gi % 7, nblk = (xcd & 3) * 4 + gi / 7;
  const int mg = (xcd >> 2) * 7 + mi7;
  const int mblk = (2 * (rank + 1) + mg) & 15;
  const int src = mblk >> 1;
  unsigned* flagp = (unsigned*)(myws + WS_FLAGS + (size_t)src * 256);
  run_tile<4, 16>(smem, myws + xoff + (size_t)mblk * BM * (GK * 2), a.w + (size_t)nblk * BN * 2, a.out + ((size_t)mblk * BM * NL + nblk * BN) * 2,
                 a.x + (size_t)mg * 16 * (GK * 2), a.out + ((size_t)(rank * MS + mg * 16) * NL + nblk * BN) * 2,
                 true, flagp, tmop, call, 0u, wid, lane TSARGS);
}

extern "C" void kernel_launch(int rank, int nranks, ncclComm_t comm,
                              hipStream_t stream,
                              void* const* d_in, const int* in_sizes, int n_in,
                              void* d_out, int out_size,
                              const int* in_dtypes, int out_dtype,
                              void* const* d_out_all,
                              void* const* d_ws_all, size_t ws_bytes) {
  if (nranks != NR || ws_bytes < WS_NEED || (unsigned)rank >= (unsigned)NR) { fprintf(stderr, "unexpected configuration\n"); return; }
  static bool init = false;
  if (!init) {
    HIP_CHECK(hipFuncSetAttribute((const void*)ag_gemm_kernel, hipFuncAttributeMaxDynamicSharedMemorySize, LDS_BYTES));
    init = true;
  }
  Args a;
  a.x = (const char*)d_in[0]; a.w = (const char*)d_in[1]; a.out = (char*)d_out; a.rank = rank;
  for (int p = 0; p < NR; ++p) a.ws[p] = (char*)d_ws_all[p];
  ag_gemm_kernel<<<NWG, 512, LDS_BYTES, stream>>>(a);
  HIP_CHECK(hipGetLastError());
}
```

```cpp
#include <hip/hip_runtime.h>
#include <rccl/rccl.h>
#include <cstdio>
#include <cstdint>

#define HIP_CHECK(expr) do { hipError_t _e = (expr); if (_e != hipSuccess) { \
    fprintf(stderr, "HIP error %s at %s:%d\n", hipGetErrorString(_e), __FILE__, __LINE__); \
    return; } } while (0)

typedef __attribute__((ext_vector_type(8))) short bf16x8;
typedef __attribute__((ext_vector_type(4))) short s16x4;
typedef __attribute__((ext_vector_type(4))) float f32x4;
typedef __attribute__((ext_vector_type(4))) unsigned u32x4;
typedef __attribute__((ext_vector_type(2))) unsigned u32x2;

constexpr int NR = 8;
constexpr int GM = 4096, GK = 4096, NL = 2048, MS = 512;
constexpr int BM = 256, BN = 128, BK = 64, NT = GK / BK;
#define NCHUNK 32
constexpr int NCH = NCHUNK, KC = GK / NCH, IPC = KC / BK;
constexpr int NWG = 256, NCW = 16;
constexpr int NSTAGE = 3;
constexpr int FJ = 3;
constexpr int A_BYTES = BM * BK * 2, B_BYTES = BK * BN * 2, ST_BYTES = A_BYTES + B_BYTES;
constexpr int CTRL_OFF = NSTAGE * (ST_BYTES + 4096);
constexpr int LDS_BYTES = CTRL_OFF + 64;

constexpr size_t XBUF_BYTES = (size_t)GM * GK * 2;
constexpr size_t WS_CTRL = 2 * XBUF_BYTES;
constexpr size_t WS_FLAGS = WS_CTRL;
constexpr size_t WS_TICK = WS_CTRL + 16384;
constexpr size_t WS_TMO = WS_CTRL + 24576;
constexpr size_t WS_HEAD = WS_CTRL + 24576 + 1024;
constexpr size_t WS_CALL = WS_CTRL + 32768;
constexpr size_t WS_DBG = WS_CALL + (size_t)NWG * 128;
constexpr size_t WS_NEED = WS_DBG + (size_t)NWG * 64;
#define TS(k) do {} while (0)
constexpr unsigned long long DEADLINE_TICKS = 50000000ull;

struct Args {
  const char* x; const char* w; char* out; char* ws[NR]; int rank;
};

__device__ __forceinline__ int ldsA_byte(int r, int c) {
  int st = (r >> 4) * 2 + (c >> 5), ob = (r & 15) * 64 + (c & 31) * 2;
  return st * 1024 + (ob ^ (((ob >> 9) & 1) << 5));
}
__device__ __forceinline__ int swzB(int row) { return ((row & 3) << 2) | ((row >> 2) & 3); }

__device__ __forceinline__ float gelu_f(float y) {
  const float c2 = 2.0f * 0.7978845608028654f;
  float u = c2 * (y + 0.044715f * y * y * y);
  return y * __builtin_amdgcn_rcpf(1.0f + __expf(-u));
}
__device__ __forceinline__ unsigned pack_bf16(float a, float b) {
  typedef __attribute__((ext_vector_type(2))) __bf16 b2;
  b2 v; v[0] = (__bf16)a; v[1] = (__bf16)b;
  return __builtin_bit_cast(unsigned, v);
}

#define LDS_ADDR(p) ((unsigned)(size_t)(__attribute__((address_space(3))) char*)(p))

template <int MI, int XR>
__device__ __forceinline__ void run_tile(char* smem, const char* Abase, const char* Bbase, char* obase, const char* Xbase, char* xobase, bool gated,
                                         unsigned* flagp, unsigned* tmop, unsigned call, unsigned cnt0, int wid, int lane
                                         ) {
  constexpr int EX = XR ? 1 : 0;
  const bool hasx = XR == 32 || (XR == 16 && wid < 4);
  constexpr int NLD = MI + 2 + EX;
  constexpr int AB = MI * 8192 + XR * 128, STB = AB + B_BYTES, NS = CTRL_OFF / STB, D = NS - 1;
  unsigned aoff[MI], boff[2];
#pragma unroll
  for (int i = 0; i < MI; ++i) {
    int bb = (wid * MI + i) * 1024 + lane * 16;
    int st = bb >> 10, sb = bb & 1023, sw = sb ^ (((sb >> 9) & 1) << 5);
    int R = (st >> 1) * 16 + (sw >> 6), C = (st & 1) * 32 + ((sw & 63) >> 1);
    aoff[i] = (unsigned)(R * (GK * 2) + C * 2);
  }
#pragma unroll
  for (int i = 0; i < 2; ++i) {
    int bb = (wid * 2 + i) * 1024 + lane * 16;
    int row = bb >> 8, pos = (bb >> 4) & 15, ch = pos ^ swzB(row);
    boff[i] = (unsigned)(row * (NL * 2) + ch * 16);
  }
  unsigned xo = 0;
  if (EX) {
    int bb = wid * (XR * 16) + (lane & (XR - 1)) * 16;
    int st = bb >> 10, sb = bb & 1023, sw = sb ^ (((sb >> 9) & 1) << 5);
    int R = (st >> 1) * 16 + (sw >> 6), C = (st & 1) * 32 + ((sw & 63) >> 1);
    xo = (unsigned)(R * (GK * 2) + C * 2);
  }
  auto stage_part = [&](int t, int s, int j) {
    if (j < MI)
      __builtin_amdgcn_global_load_lds((const unsigned*)(Abase + (size_t)t * (BK * 2) + aoff[j < MI ? j : 0]),
                                       (unsigned*)(smem + s * STB + (wid * MI + j) * 1024 + lane * 16), 16, 0, 0);
    else if (j < MI + 2)
      __builtin_amdgcn_global_load_lds((const unsigned*)(Bbase + (size_t)t * BK * (NL * 2) + boff[j == MI ? 0 : 1]),
                                       (unsigned*)(smem + s * STB + AB + (wid * 2 + j - MI) * 1024 + lane * 16), 16, 0, 0);
    else if (EX && lane < XR)
      __builtin_amdgcn_global_load_lds((const unsigned*)(Xbase + (size_t)t * (BK * 2) + xo), (unsigned*)(smem + s * STB + MI * 8192 + wid * (XR * 16) + lane * 16), 16, 0, 0);
  };
  auto stage = [&](int t, int s) {
#pragma unroll
    for (int j = 0; j < NLD; ++j) stage_part(t, s, j);
  };
  const unsigned lds0 = LDS_ADDR(smem);
  const unsigned cnt_addr = lds0 + CTRL_OFF;
  unsigned rdy = 0;
  auto wait_chunk = [&](int c) {
    if (rdy > (unsigned)c) return;
    if (wid != 0) {
      for (;;) {
        unsigned r;
        asm volatile("ds_read_b32 %0, %1 offset:4\n\ts_waitcnt lgkmcnt(0)" : "=v"(r) : "v"(cnt_addr) : "memory");
        rdy = __builtin_amdgcn_readfirstlane(r);
        if (rdy > (unsigned)c) break;
        __builtin_amdgcn_s_sleep(1);
      }
      return;
    }
    unsigned long long t0 = __builtin_amdgcn_s_memrealtime();
    unsigned nr;
    for (unsigned n = 0;; ++n) {
      unsigned f = __hip_atomic_load(flagp + (lane & (NCH - 1)), __ATOMIC_RELAXED, __HIP_MEMORY_SCOPE_SYSTEM);
      unsigned long long m = ~__builtin_amdgcn_ballot_w64(f >= call);
      nr = m ? (unsigned)__builtin_ctzll(m) : 64u;
      if (nr > (unsigned)c) break;
      if (((n & 7) == 7 && __hip_atomic_load(tmop, __ATOMIC_RELAXED, __HIP_MEMORY_SCOPE_AGENT) == call) ||
          __builtin_amdgcn_s_memrealtime() - t0 > DEADLINE_TICKS) {
        __hip_atomic_store(tmop, call, __ATOMIC_RELAXED, __HIP_MEMORY_SCOPE_AGENT);
        nr = 64u;
        break;
      }
      __builtin_amdgcn_s_sleep(2);
    }
    if (lane == 0) asm volatile("ds_write_b32 %0, %1 offset:4" :: "v"(cnt_addr), "v"(nr) : "memory");
    rdy = nr;
    if (c == 0) TS(2);
    if (c <= NCH / 2) TS(5);
    TS(6);
  };

  const int wm = wid >> 1, wn = wid & 1, fr = lane & 15, fq = lane >> 4;
  const int q = fr >> 2, p = lane & 3;
  const unsigned abase = lds0 + (unsigned)ldsA_byte(wm * (16 * MI) + fr, fq * 8);
  const unsigned ebase = lds0 + (unsigned)ldsA_byte(64 * MI + (wm >> 1) * 16 + fr, fq * 8);
  f32x4 eacc[2] = {(f32x4){0.f, 0.f, 0.f, 0.f}, (f32x4){0.f, 0.f, 0.f, 0.f}};
  unsigned bbase[4][2];
#pragma unroll
  for (int ni = 0; ni < 4; ++ni)
#pragma unroll
    for (int tt = 0; tt < 2; ++tt) {
      int row = 8 * fq + 4 * tt + q, ch = 2 * (wn * 4 + ni) + (p >> 1);
      bbase[ni][tt] = lds0 + AB + (unsigned)(256 * row + 16 * (ch ^ swzB(row)) + 8 * (p & 1));
    }
  f32x4 acc[MI][4];
#pragma unroll
  for (int i = 0; i < MI; ++i)
#pragma unroll
    for (int j = 0; j < 4; ++j) acc[i][j] = (f32x4){0.f, 0.f, 0.f, 0.f};

  bf16x8 af[2][MI], ef[2]; s16x4 bh[2][4][2];
  constexpr int NRD = 8 + MI + EX;
  auto read_part = [&](int kk, unsigned so, int j) {
    if (j < 8) {
      const int ni = j >> 1, tt = j & 1;
      unsigned ba = bbase[ni][tt] + so;
      if (kk == 0) asm volatile("ds_read_b64_tr_b16 %0, %1" : "=v"(bh[kk][ni][tt]) : "v"(ba) : "memory");
      else asm volatile("ds_read_b64_tr_b16 %0, %1 offset:8192" : "=v"(bh[kk][ni][tt]) : "v"(ba) : "memory");
    } else if (j < 8 + MI) {
      const int mi = j < 8 + MI ? j - 8 : 0;
      asm volatile("ds_read_b128 %0, %1 offset:%2" : "=v"(af[kk][mi]) : "v"(abase + so), "i"(mi * 2048 + kk * 1024) : "memory");
    } else if (EX && hasx) asm volatile("ds_read_b128 %0, %1 offset:%2" : "=v"(ef[kk]) : "v"(ebase + so), "i"(kk * 1024) : "memory");
  };
  auto read_frags = [&](int kk, unsigned so) {
#pragma unroll
    for (int j = 0; j < NRD; ++j) read_part(kk, so, j);
  };
  auto pin = [&](int kk) {
#pragma unroll
    for (int mi = 0; mi < MI; ++mi) asm volatile("" : "+v"(af[kk][mi]));
#pragma unroll
    for (int ni = 0; ni < 4; ++ni) asm volatile("" : "+v"(bh[kk][ni][0]), "+v"(bh[kk][ni][1]));
    if (EX) asm volatile("" : "+v"(ef[kk]));
  };
  auto mma = [&](int kk, bool ld, int tl, int sl, int j0, int j1) {
    constexpr int NM = MI * 4;
#pragma unroll
    for (int i = 0; i < NM; ++i) {
      const int mi = i >> 2, ni = i & 3;
      acc[mi][ni] = __builtin_amdgcn_mfma_f32_16x16x32_bf16(__builtin_shufflevector(bh[kk][ni][0], bh[kk][ni][1], 0, 1, 2, 3, 4, 5, 6, 7), af[kk][mi], acc[mi][ni], 0, 0, 0);
      if (ld) {
#pragma unroll
        for (int j = j0; j < j1; ++j)
          if (((j - j0) * NM) / (j1 - j0) == i) { __builtin_amdgcn_sched_barrier(0); stage_part(tl, sl, j); __builtin_amdgcn_sched_barrier(0); }
      }
    }
    if (EX && hasx) {
      if (wm & 1) {
        eacc[0] = __builtin_amdgcn_mfma_f32_16x16x32_bf16(__builtin_shufflevector(bh[kk][2][0], bh[kk][2][1], 0, 1, 2, 3, 4, 5, 6, 7), ef[kk], eacc[0], 0, 0, 0);
        eacc[1] = __builtin_amdgcn_mfma_f32_16x16x32_bf16(__builtin_shufflevector(bh[kk][3][0], bh[kk][3][1], 0, 1, 2, 3, 4, 5, 6, 7), ef[kk], eacc[1], 0, 0, 0);
      } else {
        eacc[0] = __builtin_amdgcn_mfma_f32_16x16x32_bf16(__builtin_shufflevector(bh[kk][0][0], bh[kk][0][1], 0, 1, 2, 3, 4, 5, 6, 7), ef[kk], eacc[0], 0, 0, 0);
        eacc[1] = __builtin_amdgcn_mfma_f32_16x16x32_bf16(__builtin_shufflevector(bh[kk][1][0], bh[kk][1][1], 0, 1, 2, 3, 4, 5, 6, 7), ef[kk], eacc[1], 0, 0, 0);
      }
    }
    __builtin_amdgcn_sched_barrier(0);
  };
  auto arrive = [&]() { if (lane == 0) asm volatile("ds_add_u32 %0, %1" :: "v"(cnt_addr), "v"(1u) : "memory"); };
  auto sync_stage = [&](int t) {
    const unsigned target = cnt0 + 8u * (unsigned)(t + 1);
    for (;;) {
      u32x2 v;
      asm volatile("ds_read_b64 %0, %1\n\ts_waitcnt lgkmcnt(0)" : "=v"(v) : "v"(cnt_addr) : "memory");
      if (__builtin_amdgcn_readfirstlane(v[0]) >= target) {
        unsigned r = __builtin_amdgcn_readfirstlane(v[1]);
        rdy = r > rdy ? r : rdy;
        break;
      }
    }
  };

#pragma unroll
  for (int i = 0; i < D; ++i) {
    if (gated && (i % IPC) == 0) wait_chunk(i / IPC);
    stage(i, i);
  }
  asm volatile("s_waitcnt vmcnt(%0)" :: "i"(NLD * (D - 1)) : "memory");
  arrive();
#define NL1 4
  constexpr int L1 = NL1 < NLD ? NL1 : 0, L0 = NLD - L1;
  int s = 0;
  for (int t = 0; t < NT - D; ++t) {
    sync_stage(t);
    unsigned f = 0;
    const bool probe = gated && wid == 0 && rdy < (unsigned)NCH;
    if (probe) asm volatile("global_load_dword %0, %1, off sc0 sc1" : "=v"(f) : "v"(flagp + (lane & (NCH - 1))) : "memory");
    if (gated && ((t + D) % IPC) == 0) wait_chunk((t + D) / IPC);
    int s2 = s + D; if (s2 >= NS) s2 -= NS;
    read_frags(0, (unsigned)(s * STB));
    read_frags(1, (unsigned)(s * STB));
    asm volatile("s_waitcnt lgkmcnt(%0)" :: "i"(8 + MI + (XR == 32 ? 1 : 0)) : "memory");
    pin(0);
    mma(0, true, t + D, s2, 0, L0);
    asm volatile("s_waitcnt lgkmcnt(0)" ::: "memory");
    pin(1);
    asm volatile("s_waitcnt vmcnt(%0)" :: "i"(NLD * (D - 2) + L0) : "memory");
    asm volatile("" : "+v"(f));
    if (probe) {
      unsigned long long m = ~__builtin_amdgcn_ballot_w64(f >= call);
      unsigned nr = m ? (unsigned)__builtin_ctzll(m) : 64u;
      if (nr > rdy) {
        rdy = nr;
        if (lane == 0) asm volatile("ds_write_b32 %0, %1 offset:4" :: "v"(cnt_addr), "v"(nr) : "memory");
      }
    }
    arrive();
    mma(1, L1 > 0, t + D, s2, L0, NLD);
    s = s + 1; if (s >= NS) s = 0;
  }
  for (int t = NT - D; t < NT; ++t) {
    sync_stage(t);
    read_frags(0, (unsigned)(s * STB));
    read_frags(1, (unsigned)(s * STB));
    asm volatile("s_waitcnt lgkmcnt(%0)" :: "i"(8 + MI + (XR == 32 ? 1 : 0)) : "memory");
    pin(0);
    mma(0, false, 0, 0, 0, 1);
    asm volatile("s_waitcnt lgkmcnt(0)" ::: "memory");
    pin(1);
    if (D > 2 && t + D == NT) asm volatile("s_waitcnt vmcnt(%0)" :: "i"(NLD * (D > 2 ? D - 2 : 0)) : "memory");
    else if (D > 3 && t + D == NT + 1) asm volatile("s_waitcnt vmcnt(%0)" :: "i"(NLD * (D > 3 ? D - 3 : 0)) : "memory");
    else if (D > 4 && t + D == NT + 2) asm volatile("s_waitcnt vmcnt(%0)" :: "i"(NLD * (D > 4 ? D - 4 : 0)) : "memory");
    else asm volatile("s_waitcnt vmcnt(0)" ::: "memory");
    arrive();
    mma(1, false, 0, 0, 0, 1);
    s = s + 1; if (s >= NS) s = 0;
  }
  if (wid == 0 && gated) TS(3);

  const bool bad = gated && (__hip_atomic_load(tmop, __ATOMIC_RELAXED, __HIP_MEMORY_SCOPE_AGENT) == call);
  auto store_pair = [&](char* p, const f32x4& u, const f32x4& v, bool nan) {
    unsigned a0 = pack_bf16(gelu_f(u[0]), gelu_f(u[1])), a1 = pack_bf16(gelu_f(u[2]), gelu_f(u[3]));
    unsigned b0 = pack_bf16(gelu_f(v[0]), gelu_f(v[1])), b1 = pack_bf16(gelu_f(v[2]), gelu_f(v[3]));
    auto r0 = __builtin_amdgcn_permlane16_swap(a0, b0, false, false);
    auto r1 = __builtin_amdgcn_permlane16_swap(a1, b1, false, false);
    u32x4 o = {r0[0], r1[0], r0[1], r1[1]};
    if (nan) o = (u32x4){0x7FC07FC0u, 0x7FC07FC0u, 0x7FC07FC0u, 0x7FC07FC0u};
    __builtin_nontemporal_store(o, (u32x4*)p);
  };
  const int lcol = (fq & 1) * 16 + (fq & 2) * 4;
  char* ob = obase + ((size_t)(wm * (16 * MI) + fr) * NL + wn * 64 + lcol) * 2;
#pragma unroll
  for (int mi = 0; mi < MI; ++mi)
#pragma unroll
    for (int h = 0; h < 2; ++h) store_pair(ob + ((size_t)mi * 16 * NL + h * 32) * 2, acc[mi][2 * h], acc[mi][2 * h + 1], bad);
  if (EX && hasx) store_pair(xobase + ((size_t)((wm >> 1) * 16 + fr) * NL + wn * 64 + (wm & 1) * 32 + lcol) * 2, eacc[0], eacc[1], false);
}

__global__ __launch_bounds__(512, 1) void ag_gemm_kernel(Args a) {
  extern __shared__ __attribute__((aligned(16))) char smem[];
  const int tid = threadIdx.x, lane = tid & 63;
  const int wid = __builtin_amdgcn_readfirstlane(tid >> 6);
  const int b = blockIdx.x, rank = a.rank;
  char* myws = a.ws[rank];
  const int xcd = b & 7, idx = b >> 3;
  const bool is_comm = idx < NCW / 8;
  constexpr unsigned RPI = 512 / KC, LPR = KC / 8;
  constexpr unsigned IPCH = MS / RPI, NITEMS = NCH * IPCH;
  const unsigned lo = (unsigned)(lane / LPR) * (GK * 2) + (unsigned)(lane % LPR) * 16;
  unsigned it = (unsigned)((idx * 8 + xcd) * 8 + wid);
  unsigned off = (it & (IPCH - 1)) * (RPI * GK * 2) + (it / IPCH) * (KC * 2) + lo;
  u32x4 v0;
  if (is_comm) asm volatile("global_load_dwordx4 %0, %1, off" : "=v"(v0) : "v"(a.x + off) : "memory");
  volatile unsigned* ctl = (volatile unsigned*)(smem + CTRL_OFF);
  if (tid == 0) {
    unsigned* cp = (unsigned*)(myws + WS_CALL + (size_t)b * 128);
    unsigned c = __hip_atomic_load(cp, __ATOMIC_RELAXED, __HIP_MEMORY_SCOPE_AGENT) + 1u;
    __hip_atomic_store(cp, c, __ATOMIC_RELAXED, __HIP_MEMORY_SCOPE_AGENT);
    ctl[0] = 0; ctl[1] = 0; ctl[2] = c;
    if (!is_comm) {
      __builtin_amdgcn_fence(__ATOMIC_ACQUIRE, "");
      asm volatile("s_waitcnt vmcnt(0)" ::: "memory");
    }
  }
  __syncthreads();
  const unsigned call = __builtin_amdgcn_readfirstlane(ctl[2]);
  const size_t xoff = (call & 1u) ? XBUF_BYTES : 0;
  if (wid == 0) TS(0);

  if (is_comm) {
    const unsigned base = (call - 1u) * NITEMS - NCW * 8;
    char* headp = myws + WS_HEAD;
    const size_t dbase = xoff + (size_t)rank * MS * (GK * 2);
    char* pw = a.ws[0];
#pragma unroll
    for (int p = 1; p < NR; ++p) pw = (lane == p) ? a.ws[p] : pw;
    pw += WS_FLAGS + (size_t)rank * 256;
    char* dst[NR - 1];
#pragma unroll
    for (int j = 0; j < NR - 1; ++j) dst[j] = a.ws[(rank + 1 + (j + b * 8 + wid) % 7) & 7] + dbase;
    unsigned g = 0, tk = 0, pc = 0xffffffffu;
    asm volatile("s_waitcnt vmcnt(0)" : "+v"(v0) :: "memory");
    while (it < NITEMS) {
      if (lane == 0) asm volatile("global_atomic_add %0, %1, %2, off sc0" : "=v"(g) : "v"(headp), "v"(1u) : "memory");
#pragma unroll
      for (int j = 0; j < FJ; ++j)
        asm volatile("global_store_dwordx4 %0, %1, off sc0 sc1" :: "v"(dst[j] + off), "v"(v0) : "memory");
      asm volatile("s_waitcnt vmcnt(%2)" : "+v"(g), "+v"(tk) : "i"(FJ) : "memory");
      if (pc != 0xffffffffu && (__builtin_amdgcn_readfirstlane(tk) & (IPCH - 1)) == IPCH - 1 && lane < NR && lane != rank)
        asm volatile("global_store_dword %0, %1, off sc0 sc1" :: "v"(pw + (size_t)pc * 4), "v"(call) : "memory");
#pragma unroll
      for (int j = FJ; j < NR - 1; ++j)
        asm volatile("global_store_dwordx4 %0, %1, off sc0 sc1" :: "v"(dst[j] + off), "v"(v0) : "memory");
      pc = it / IPCH;
      it = __builtin_amdgcn_readfirstlane(g) - base;
      off = (it & (IPCH - 1)) * (RPI * GK * 2) + (it / IPCH) * (KC * 2) + lo;
      if (it < NITEMS) asm volatile("global_load_dwordx4 %0, %1, off" : "=v"(v0) : "v"(a.x + off) : "memory");
      asm volatile("s_waitcnt vmcnt(0)" : "+v"(v0) :: "memory");
      if (lane == 0) asm volatile("global_atomic_add %0, %1, %2, off sc0" : "=v"(tk) : "v"(myws + WS_TICK + (size_t)pc * 128), "v"(1u) : "memory");
    }
    asm volatile("s_waitcnt vmcnt(0)" : "+v"(tk) :: "memory");
    if (pc != 0xffffffffu && (__builtin_amdgcn_readfirstlane(tk) & (IPCH - 1)) == IPCH - 1 && lane < NR && lane != rank)
      asm volatile("global_store_dword %0, %1, off sc0 sc1" :: "v"(pw + (size_t)pc * 4), "v"(call) : "memory");
    if (wid == 0) TS(1);
    return;
  }

  unsigned* tmop = (unsigned*)(myws + WS_TMO);
#define TSARGS
  if (idx < NCW / 8 + 2) {
    const int nblk = xcd * 2 + (idx - NCW / 8);
    run_tile<4, 32>(smem, a.x + (size_t)224 * (GK * 2), a.w + (size_t)nblk * BN * 2, a.out + ((size_t)(rank * MS + 224) * NL + nblk * BN) * 2,
                    a.x + (size_t)480 * (GK * 2), a.out + ((size_t)(rank * MS + 480) * NL + nblk * BN) * 2, false, nullptr, tmop, call, 0u, wid, lane TSARGS);
    if (wid == 0) TS(7);
    return;
  }
  const int gi = idx - (NCW / 8 + 2);
  const int mi7 = gi % 7, nblk = (xcd & 3) * 4 + gi / 7;
  const int mg = (xcd >> 2) * 7 + mi7;
  const int mblk = (2 * (rank + 1) + mg) & 15;
  const int src = mblk >> 1;
  unsigned* flagp = (unsigned*)(myws + WS_FLAGS + (size_t)src * 256);
  run_tile<4, 16>(smem, myws + xoff + (size_t)mblk * BM * (GK * 2), a.w + (size_t)nblk * BN * 2, a.out + ((size_t)mblk * BM * NL + nblk * BN) * 2,
                 a.x + (size_t)mg * 16 * (GK * 2), a.out + ((size_t)(rank * MS + mg * 16) * NL + nblk * BN) * 2,
                 true, flagp, tmop, call, 0u, wid, lane TSARGS);
}

extern "C" void kernel_launch(int rank, int nranks, ncclComm_t comm,
                              hipStream_t stream,
                              void* const* d_in, const int* in_sizes, int n_in,
                              void* d_out, int out_size,
                              const int* in_dtypes, int out_dtype,
                              void* const* d_out_all,
                              void* const* d_ws_all, size_t ws_bytes) {
  if (nranks != NR || ws_bytes < WS_NEED || (unsigned)rank >= (unsigned)NR) { fprintf(stderr, "unexpected configuration\n"); return; }
  static bool init = false;
  if (!init) {
    HIP_CHECK(hipFuncSetAttribute((const void*)ag_gemm_kernel, hipFuncAttributeMaxDynamicSharedMemorySize, LDS_BYTES));
    init = true;
  }
  Args a;
  a.x = (const char*)d_in[0]; a.w = (const char*)d_in[1]; a.out = (char*)d_out; a.rank = rank;
  for (int p = 0; p < NR; ++p) a.ws[p] = (char*)d_ws_all[p];
  ag_gemm_kernel<<<NWG, 512, LDS_BYTES, stream>>>(a);
  HIP_CHECK(hipGetLastError());
}
```
